```python
import numpy as np

kernel_name = "gpu_a2a_gemm_m4096_k4096_n16384_f32_silu"


ABSMAX_RTOL = 0.001

COMPUTE_DTYPE = "float32"
EPILOGUE = "silu"
IN_QUANT = None
OUT_QUANT = None
IN_DTYPES = {"x": "float32", "w_mat": "float32"}
WORLD_SIZE = 8

def setup_inputs():

    x = np.random.randn(4096, 4096).astype(np.float32)
    w_mat = (np.random.randn(4096, 16384) / np.sqrt(4096)).astype(np.float32)
    return {"x": x, "w_mat": w_mat}

def _epilogue(y):
    return (y / (1.0 + np.exp(-np.clip(y, -60.0, 60.0)))).astype(np.float32)

def reference(x, w_mat):
    y = (x @ w_mat).astype(np.float32)
    return _epilogue(y)
```

```cpp
#include <hip/hip_runtime.h>
#include <rccl/rccl.h>
#include <cstdio>

namespace {
constexpr unsigned kM = 512, kK = 4096, kN = 16384, kRanks = 8;
constexpr unsigned kSlotBytes = kM * kK * 2;
constexpr unsigned kCtl = 2 * kSlotBytes;
constexpr unsigned kCntOff = 4096;
constexpr unsigned kOwnOff = 32768;
constexpr unsigned kFlagOff = 65536;
constexpr unsigned kCallOff = 262144;
constexpr unsigned kCtlBytes = 1u << 20;
constexpr unsigned kLdsBytes = 2 * 16384 + 64;
constexpr unsigned kSrd3 = 0x00020000u;

struct Params {
  const float* x; const float* w; float* out;
  char* ws[kRanks];
  unsigned rank, pad;
};
typedef unsigned long long u64;
typedef unsigned u32x4 __attribute__((ext_vector_type(4)));
typedef unsigned u32x8 __attribute__((ext_vector_type(8)));
}

__global__ __launch_bounds__(256, 1) void a2a_gemm_silu_kernel(const Params p) {
  __shared__ __attribute__((aligned(16))) char smem[kLdsBytes];
  unsigned b = blockIdx.x;
  const unsigned lane = threadIdx.x & 63, w = __builtin_amdgcn_readfirstlane(threadIdx.x >> 6), rank = p.rank & 7u;
  const u64 wsme = (u64)p.ws[7], cb = wsme + kCtl, xb = (u64)p.x;
  unsigned s60 = (unsigned)xb, s61 = (unsigned)(xb >> 32), s62 = kK * kM * 4, s63 = kSrd3;
  unsigned s72 = (unsigned)cb, s73 = (unsigned)(cb >> 32), s74 = kCtlBytes, s75 = kSrd3;
  auto frag = [&](unsigned g, unsigned s, unsigned& q, unsigned& j, unsigned& cnt) {
    const unsigned R = 2 * s + (g >> 9), blk = (R - 1) & 7, e = (g >> 6) & 7, f = g & 63;
    q = R == 0 ? 7u : (e < 7 ? e : blk); j = R == 0 ? e : (e < 7 ? blk : 7u);
    cnt = (R == 0 ? kOwnOff : kCntOff + blk * 2048) + e * 256 + (f >> 5) * 128;
    return (((((rank - 1 - q) & 7u) * 512 + (f & 31) * 16) * 512 + j * 64 + (f >> 5) * 32) * 4);
  };
  unsigned q0, j0, n0;
  unsigned s76 = frag(b * 4 + w, 0, q0, j0, n0), s96 = kCallOff + (b * 4 + w) * 128;
  const unsigned xoff = (lane & 15) * 2048 + (lane >> 4) * 32;
  const unsigned fp = w == 2 ? (b >> 3) & 1 : w >> 1;
  const unsigned prow = w == 2 ? ((b >> 6) & 1) * 256 + (b & 63) * 4 : ((b >> 3) & 15) * 256 + ((b & 7) + (b >> 7) * 8) * 16 + w * 4;
  u64 prb = (w == 2 ? (u64)p.out : (u64)p.w) + prow * (kN * 4) + 16384 + (lane & 7) * 1024;
  prb ^= (u64)(((unsigned)(prb >> 13) ^ (unsigned)(prb >> 21) ^ fp) & 1u) << 13;
  unsigned s54 = (w && (b >> 7)) ? 4 + (((b >> 2) ^ fp) & 1) : (w & 1) ? ((b >> 2) ^ fp) & 1 : 8u;
  unsigned c19, c25; u32x4 c20; u32x8 c160; u64 c30;
  asm volatile(
    "s_memrealtime s[30:31]\n"
    "v_mov_b32 v19, 1\n"
    "v_xor_b32 v20, 0x2000, v26\n"
    "v_mov_b32 v21, v27\n"
    "s_bitcmp1_b32 s54, 2\n"
    "s_cbranch_scc0 LATE%=\n"
    "s_mov_b64 exec, 0xff\n"
    "global_load_dword v22, v[26:27], off nt\n"
    "global_load_dword v23, v[20:21], off nt\n"
    "s_mov_b64 exec, -1\n"
    "s_mov_b64 exec, 1\n"
    "buffer_atomic_add v19, off, s[72:75], s96 sc0\n"
    "s_mov_b64 exec, -1\n"
    "buffer_load_dwordx4 v[160:163], v9, s[60:63], s76 offen offset:0 nt\n"
    "buffer_load_dwordx4 v[164:167], v9, s[60:63], s76 offen offset:16 nt\n"
    "buffer_load_dword v25, off, s[72:75], s96 offset:4 sc1\n"
    "s_waitcnt vmcnt(5)\n"
    "s_memtime s[48:49]\n"
    "s_waitcnt lgkmcnt(0)\n"
    "s_mov_b32 s52, s48\n"
    "s_waitcnt vmcnt(4)\n"
    "s_memtime s[48:49]\n"
    "s_waitcnt lgkmcnt(0)\n"
    "s_sub_u32 s52, s48, s52\n"
    "s_and_b32 s54, s54, 1\n"
    "s_lshl_b32 s54, s54, 1\n"
    "s_cmp_gt_u32 s52, 64\n"
    "s_cselect_b32 s54, s54, 1\n"
    "v_mov_b32 v20, 1\n"
    "v_mov_b32 v21, s54\n"
    "s_mov_b32 s54, 2\n"
    "s_branch ISSUED%=\n"
    "LATE%=:\n"
    "s_mov_b64 exec, 1\n"
    "buffer_atomic_add v19, off, s[72:75], s96 sc0\n"
    "s_mov_b64 exec, -1\n"
    "buffer_load_dwordx4 v[160:163], v9, s[60:63], s76 offen offset:0 nt\n"
    "buffer_load_dwordx4 v[164:167], v9, s[60:63], s76 offen offset:16 nt\n"
    "buffer_load_dword v25, off, s[72:75], s96 offset:4 sc1\n"
    "s_bitcmp1_b32 s54, 3\n"
    "s_cbranch_scc1 ISSUED%=\n"
    "s_mov_b64 exec, 0xff\n"
    "global_load_dword v22, v[26:27], off nt\n"
    "global_load_dword v23, v[20:21], off nt\n"
    "s_mov_b64 exec, -1\n"
    "ISSUED%=:\n"
    : "={v19}"(c19), "={v25}"(c25), "={v[20:23]}"(c20), "={v[160:167]}"(c160), "={s[30:31]}"(c30), "+{s54}"(s54), "+s"(b)
    : "{s[60:63]}"(u32x4{s60, s61, s62, s63}), "{s[72:75]}"(u32x4{s72, s73, s74, s75}), "{s76}"(s76), "{s96}"(s96), "{v9}"(xoff), "{v[26:27]}"(prb)
    : "s48", "s49", "s52", "scc", "memory");
  const unsigned n = w & 1, m = w >> 1, role = (b >> 2) & 1, g = b * 4 + w, hi = b >> 7;
  const unsigned nt = ((b >> 1) & 1) | ((b >> 2) & 14) | ((b & 1) << 4) | ((b >> 1) & 96);
  const unsigned ldsb = (unsigned)(size_t)smem;
  const unsigned woffb = (nt * 128 + n * 64) * 4;
  const u64 ot = (u64)p.out + nt * 512;
  const unsigned par = (role ^ (unsigned)(ot >> 13) ^ (unsigned)(ot >> 21)) & 1;
  const unsigned ooffb = ((w * 128 + par * 32) * kN + nt * 128) * 4;
  const unsigned poffb = ((w * 128 + (par ^ 1) * 32) * kN + nt * 128) * 4;
  const u64 wb = (u64)p.w + woffb, ob = (u64)p.out + ooffb, pb = (u64)p.out + poffb;
  unsigned s36 = (unsigned)wb, s37 = (unsigned)(wb >> 32), s38 = kK * kN * 4 - woffb, s39 = kSrd3;
  unsigned s40 = (unsigned)wsme, s41 = (unsigned)(wsme >> 32), s42 = kSlotBytes, s43 = kSrd3;
  unsigned s64 = (unsigned)ob, s65 = (unsigned)(ob >> 32), s66 = kM * kN * 4 - ooffb, s67 = kSrd3;
  unsigned s68 = (unsigned)pb, s69 = (unsigned)(pb >> 32), s70 = kM * kN * 4 - poffb, s71 = kSrd3;
  unsigned xo[4]; u64 dst[4], ca[4];
#pragma unroll
  for (unsigned s = 0; s < 4; ++s) {
    unsigned q, j, cnt;
    xo[s] = frag(g, s, q, j, cnt);
    const u64 wd = (u64)p.ws[q & 7u];
    dst[s] = wd + (rank * 8 + j) * 65536 + (g & 63) * 1024;
    ca[s] = wd + kCtl + cnt;
  }
  unsigned s77 = xo[1], s78 = xo[2], s79 = xo[3];
  unsigned s80 = (unsigned)dst[0], s81 = (unsigned)(dst[0] >> 32), s82 = (unsigned)dst[1], s83 = (unsigned)(dst[1] >> 32);
  unsigned s84 = (unsigned)dst[2], s85 = (unsigned)(dst[2] >> 32), s86 = (unsigned)dst[3], s87 = (unsigned)(dst[3] >> 32);
  unsigned s88 = (unsigned)ca[0], s89 = (unsigned)(ca[0] >> 32), s90 = (unsigned)ca[1], s91 = (unsigned)(ca[1] >> 32);
  unsigned s92 = (unsigned)ca[2], s93 = (unsigned)(ca[2] >> 32), s94 = (unsigned)ca[3], s95 = (unsigned)(ca[3] >> 32);
  unsigned s97 = kFlagOff + g * 128, s98 = kFlagOff + ((b ^ 4) * 4 + w) * 128;
  unsigned s55 = m, s58 = hi * 4, s51 = w, s53 = hi ? 1u : 0xffu;
  unsigned s99 = 256 + (w == 2 ? 16 + ((b >> 6) & 1) * 2 + (b & 1) : (b >> 3) & 15) * 128;
  const unsigned ti = (lane - 8) >> 3, te = lane & 7;
  const unsigned ks = lane < 8 ? rank * 8 + lane : (te < 7 ? ((rank + 1 + te) & 7) * 8 + ti : ((rank + 1 + ti) & 7) * 8 + 7);
  const u64 tb = (u64)p.w + nt * 512;
  const unsigned side = (role ^ (unsigned)(tb >> 13) ^ (unsigned)(tb >> 21)) & 1;
  unsigned tab = ks * 2 + (lane < 4 ? (role ^ lane) & 1 : side);
  const unsigned woff = (lane >> 4) * 8 * (kN * 4) + (lane & 15) * 16;
  unsigned alo = w * 8192 + par * 2048 + lane * 16, ahi = alo ^ 2048;
  const unsigned br = ldsb + lane * 16, bw = br + m * 8192 + n * 4096;
  const unsigned ooff = (lane >> 4) * 4 * (kN * 4) + (lane & 15) * 16;
  const unsigned l16 = lane * 16, poff = (lane & 15) * 128;
  const unsigned poff0 = lane - 16 < 20 ? 256 + (lane - 16) * 128 : kOwnOff + (b & 7) * 4096 + poff;
  asm volatile(
    "s_mov_b32 s44, s38\n"
    "s_mov_b32 s45, s42\n"
    "s_mov_b32 s59, 1.0\n"
    "v_mov_b32 v15, 0\n"
    "v_mov_b32 v16, 4\n"
    "s_nop 0\n"
    "v_readlane_b32 s47, v2, s55\n"
    "s_cmp_lt_u32 s54, 2\n"
    "s_cbranch_scc1 PROBING%=\n"
    "s_waitcnt vmcnt(0)\n"
    "PROBING%=:\n"
    "s_waitcnt vmcnt(2) lgkmcnt(0)\n"
    "v_readfirstlane_b32 s57, v19\n"
    "v_cvt_pk_f16_f32 v40, v160, v161\n"
    "v_cvt_pk_f16_f32 v41, v162, v163\n"
    "v_cvt_pk_f16_f32 v42, v164, v165\n"
    "v_cvt_pk_f16_f32 v43, v166, v167\n"
    "s_add_u32 s57, s57, 1\n"
    "s_and_b32 s56, s57, 1\n"
    "s_lshl_b32 s56, s56, 22\n"
    "s_add_u32 s80, s80, s56\n"
    "s_addc_u32 s81, s81, 0\n"
    "global_store_dwordx4 v10, v[40:43], s[80:81] sc0 sc1\n"
    "s_cmp_lt_u32 s54, 2\n"
    "s_cbranch_scc0 NOTIME%=\n"
    "s_waitcnt vmcnt(2)\n"
    "s_memtime s[48:49]\n"
    "s_waitcnt lgkmcnt(0)\n"
    "s_mov_b32 s52, s48\n"
    "s_waitcnt vmcnt(1)\n"
    "s_memtime s[48:49]\n"
    "s_waitcnt lgkmcnt(0)\n"
    "s_sub_u32 s52, s48, s52\n"
    "s_and_b32 s54, s54, 1\n"
    "s_lshl_b32 s54, s54, 1\n"
    "s_cmp_gt_u32 s52, 64\n"
    "s_cselect_b32 s54, s54, 1\n"
    "v_mov_b32 v20, 1\n"
    "v_mov_b32 v21, s54\n"
    "s_mov_b32 s54, 2\n"
    "NOTIME%=:\n"
    "s_lshl_b32 s48, s47, 21\n"
    "buffer_load_dwordx4 v[192:195], v3, s[36:39], s48 offen nt\n"
    "s_add_u32 s48, s48, 0x10000\n"
    "buffer_load_dwordx4 v[196:199], v3, s[36:39], s48 offen nt\n"
    "s_add_u32 s48, s48, 0x10000\n"
    "buffer_load_dwordx4 v[200:203], v3, s[36:39], s48 offen nt\n"
    "s_add_u32 s48, s48, 0x10000\n"
    "buffer_load_dwordx4 v[204:207], v3, s[36:39], s48 offen nt\n"
    "s_add_u32 s48, s48, 0x10000\n"
    "s_cmp_eq_u32 s54, 2\n"
    "s_cbranch_scc0 NOVOTE%=\n"
    "s_mov_b64 exec, 1\n"
    "buffer_atomic_add_x2 v[20:21], off, s[72:75], s99\n"
    "s_mov_b64 exec, -1\n"
    "NOVOTE%=:\n"
    "s_add_u32 s82, s82, s56\n"
    "s_addc_u32 s83, s83, 0\n"
    "s_add_u32 s84, s84, s56\n"
    "s_addc_u32 s85, s85, 0\n"
    "s_add_u32 s86, s86, s56\n"
    "s_addc_u32 s87, s87, 0\n"
    "s_add_u32 s40, s40, s56\n"
    "s_addc_u32 s41, s41, 0\n"
    "s_lshl_b32 s56, s57, 5\n"
    "v_mov_b32 v18, s56\n"
    "v_mov_b32 v17, s56\n"
    "v_accvgpr_write_b32 a0, 0\n"
    "v_accvgpr_write_b32 a1, 0\n"
    "v_accvgpr_write_b32 a2, 0\n"
    "v_accvgpr_write_b32 a3, 0\n"
    "v_accvgpr_write_b32 a4, 0\n"
    "v_accvgpr_write_b32 a5, 0\n"
    "v_accvgpr_write_b32 a6, 0\n"
    "v_accvgpr_write_b32 a7, 0\n"
    "v_accvgpr_write_b32 a8, 0\n"
    "v_accvgpr_write_b32 a9, 0\n"
    "v_accvgpr_write_b32 a10, 0\n"
    "v_accvgpr_write_b32 a11, 0\n"
    "v_accvgpr_write_b32 a12, 0\n"
    "v_accvgpr_write_b32 a13, 0\n"
    "v_accvgpr_write_b32 a14, 0\n"
    "v_accvgpr_write_b32 a15, 0\n"
    "v_accvgpr_write_b32 a16, 0\n"
    "v_accvgpr_write_b32 a17, 0\n"
    "v_accvgpr_write_b32 a18, 0\n"
    "v_accvgpr_write_b32 a19, 0\n"
    "v_accvgpr_write_b32 a20, 0\n"
    "v_accvgpr_write_b32 a21, 0\n"
    "v_accvgpr_write_b32 a22, 0\n"
    "v_accvgpr_write_b32 a23, 0\n"
    "v_accvgpr_write_b32 a24, 0\n"
    "v_accvgpr_write_b32 a25, 0\n"
    "v_accvgpr_write_b32 a26, 0\n"
    "v_accvgpr_write_b32 a27, 0\n"
    "v_accvgpr_write_b32 a28, 0\n"
    "v_accvgpr_write_b32 a29, 0\n"
    "v_accvgpr_write_b32 a30, 0\n"
    "v_accvgpr_write_b32 a31, 0\n"
    "v_accvgpr_write_b32 a32, 0\n"
    "v_accvgpr_write_b32 a33, 0\n"
    "v_accvgpr_write_b32 a34, 0\n"
    "v_accvgpr_write_b32 a35, 0\n"
    "v_accvgpr_write_b32 a36, 0\n"
    "v_accvgpr_write_b32 a37, 0\n"
    "v_accvgpr_write_b32 a38, 0\n"
    "v_accvgpr_write_b32 a39, 0\n"
    "v_accvgpr_write_b32 a40, 0\n"
    "v_accvgpr_write_b32 a41, 0\n"
    "v_accvgpr_write_b32 a42, 0\n"
    "v_accvgpr_write_b32 a43, 0\n"
    "v_accvgpr_write_b32 a44, 0\n"
    "v_accvgpr_write_b32 a45, 0\n"
    "v_accvgpr_write_b32 a46, 0\n"
    "v_accvgpr_write_b32 a47, 0\n"
    "v_accvgpr_write_b32 a48, 0\n"
    "v_accvgpr_write_b32 a49, 0\n"
    "v_accvgpr_write_b32 a50, 0\n"
    "v_accvgpr_write_b32 a51, 0\n"
    "v_accvgpr_write_b32 a52, 0\n"
    "v_accvgpr_write_b32 a53, 0\n"
    "v_accvgpr_write_b32 a54, 0\n"
    "v_accvgpr_write_b32 a55, 0\n"
    "v_accvgpr_write_b32 a56, 0\n"
    "v_accvgpr_write_b32 a57, 0\n"
    "v_accvgpr_write_b32 a58, 0\n"
    "v_accvgpr_write_b32 a59, 0\n"
    "v_accvgpr_write_b32 a60, 0\n"
    "v_accvgpr_write_b32 a61, 0\n"
    "v_accvgpr_write_b32 a62, 0\n"
    "v_accvgpr_write_b32 a63, 0\n"
    "v_accvgpr_write_b32 a64, 0\n"
    "v_accvgpr_write_b32 a65, 0\n"
    "v_accvgpr_write_b32 a66, 0\n"
    "v_accvgpr_write_b32 a67, 0\n"
    "v_accvgpr_write_b32 a68, 0\n"
    "v_accvgpr_write_b32 a69, 0\n"
    "v_accvgpr_write_b32 a70, 0\n"
    "v_accvgpr_write_b32 a71, 0\n"
    "v_accvgpr_write_b32 a72, 0\n"
    "v_accvgpr_write_b32 a73, 0\n"
    "v_accvgpr_write_b32 a74, 0\n"
    "v_accvgpr_write_b32 a75, 0\n"
    "v_accvgpr_write_b32 a76, 0\n"
    "v_accvgpr_write_b32 a77, 0\n"
    "v_accvgpr_write_b32 a78, 0\n"
    "v_accvgpr_write_b32 a79, 0\n"
    "v_accvgpr_write_b32 a80, 0\n"
    "v_accvgpr_write_b32 a81, 0\n"
    "v_accvgpr_write_b32 a82, 0\n"
    "v_accvgpr_write_b32 a83, 0\n"
    "v_accvgpr_write_b32 a84, 0\n"
    "v_accvgpr_write_b32 a85, 0\n"
    "v_accvgpr_write_b32 a86, 0\n"
    "v_accvgpr_write_b32 a87, 0\n"
    "v_accvgpr_write_b32 a88, 0\n"
    "v_accvgpr_write_b32 a89, 0\n"
    "v_accvgpr_write_b32 a90, 0\n"
    "v_accvgpr_write_b32 a91, 0\n"
    "v_accvgpr_write_b32 a92, 0\n"
    "v_accvgpr_write_b32 a93, 0\n"
    "v_accvgpr_write_b32 a94, 0\n"
    "v_accvgpr_write_b32 a95, 0\n"
    "v_accvgpr_write_b32 a96, 0\n"
    "v_accvgpr_write_b32 a97, 0\n"
    "v_accvgpr_write_b32 a98, 0\n"
    "v_accvgpr_write_b32 a99, 0\n"
    "v_accvgpr_write_b32 a100, 0\n"
    "v_accvgpr_write_b32 a101, 0\n"
    "v_accvgpr_write_b32 a102, 0\n"
    "v_accvgpr_write_b32 a103, 0\n"
    "v_accvgpr_write_b32 a104, 0\n"
    "v_accvgpr_write_b32 a105, 0\n"
    "v_accvgpr_write_b32 a106, 0\n"
    "v_accvgpr_write_b32 a107, 0\n"
    "v_accvgpr_write_b32 a108, 0\n"
    "v_accvgpr_write_b32 a109, 0\n"
    "v_accvgpr_write_b32 a110, 0\n"
    "v_accvgpr_write_b32 a111, 0\n"
    "v_accvgpr_write_b32 a112, 0\n"
    "v_accvgpr_write_b32 a113, 0\n"
    "v_accvgpr_write_b32 a114, 0\n"
    "v_accvgpr_write_b32 a115, 0\n"
    "v_accvgpr_write_b32 a116, 0\n"
    "v_accvgpr_write_b32 a117, 0\n"
    "v_accvgpr_write_b32 a118, 0\n"
    "v_accvgpr_write_b32 a119, 0\n"
    "v_accvgpr_write_b32 a120, 0\n"
    "v_accvgpr_write_b32 a121, 0\n"
    "v_accvgpr_write_b32 a122, 0\n"
    "v_accvgpr_write_b32 a123, 0\n"
    "v_accvgpr_write_b32 a124, 0\n"
    "v_accvgpr_write_b32 a125, 0\n"
    "v_accvgpr_write_b32 a126, 0\n"
    "v_accvgpr_write_b32 a127, 0\n"
    "s_cmp_eq_u32 s54, 2\n"
    "s_cbranch_scc1 ACKV%=\n"
    "s_waitcnt vmcnt(4)\n"
    "ACKV%=:\n"
    "s_waitcnt vmcnt(5)\n"
    "s_barrier\n"
    "s_cmp_eq_u32 s51, 0\n"
    "s_cbranch_scc0 NOSIG%=\n"
    "s_mov_b32 exec_lo, s53\n"
    "s_mov_b32 exec_hi, 0\n"
    "global_atomic_add v12, v16, s[88:89] sc1\n"
    "s_mov_b64 exec, -1\n"
    "NOSIG%=:\n"
    "s_add_u32 s55, s55, 2\n"
    "v_readlane_b32 s47, v2, s55\n"
    "buffer_load_dwordx4 v[208:211], v3, s[36:39], s48 offen nt\n"
    "s_add_u32 s48, s48, 0x10000\n"
    "buffer_load_dwordx4 v[212:215], v3, s[36:39], s48 offen nt\n"
    "s_add_u32 s48, s48, 0x10000\n"
    "buffer_load_dwordx4 v[216:219], v3, s[36:39], s48 offen nt\n"
    "s_add_u32 s48, s48, 0x10000\n"
    "buffer_load_dwordx4 v[220:223], v3, s[36:39], s48 offen nt\n"
    "s_add_u32 s48, s48, 0x10000\n"
    "s_lshl_b32 s48, s47, 21\n"
    "buffer_load_dwordx4 v[224:227], v3, s[36:39], s48 offen nt\n"
    "s_add_u32 s48, s48, 0x10000\n"
    "buffer_load_dwordx4 v[228:231], v3, s[36:39], s48 offen nt\n"
    "s_add_u32 s48, s48, 0x10000\n"
    "buffer_load_dwordx4 v[232:235], v3, s[36:39], s48 offen nt\n"
    "s_add_u32 s48, s48, 0x10000\n"
    "buffer_load_dwordx4 v[236:239], v3, s[36:39], s48 offen nt\n"
    "s_add_u32 s48, s48, 0x10000\n"
    "buffer_load_dwordx4 v[240:243], v3, s[36:39], s48 offen nt\n"
    "s_add_u32 s48, s48, 0x10000\n"
    "buffer_load_dwordx4 v[244:247], v3, s[36:39], s48 offen nt\n"
    "s_add_u32 s48, s48, 0x10000\n"
    "buffer_load_dwordx4 v[248:251], v3, s[36:39], s48 offen nt\n"
    "s_add_u32 s48, s48, 0x10000\n"
    "buffer_load_dwordx4 v[252:255], v3, s[36:39], s48 offen nt\n"
    "s_add_u32 s48, s48, 0x10000\n"
    "v_readlane_b32 s52, v2, 0\n"
    "v_readlane_b32 s46, v2, 1\n"
    "v_accvgpr_write_b32 a128, 0\n"
    "v_accvgpr_write_b32 a129, 0\n"
    "v_accvgpr_write_b32 a130, 0\n"
    "v_accvgpr_write_b32 a131, 0\n"
    "v_accvgpr_write_b32 a132, 0\n"
    "v_accvgpr_write_b32 a133, 0\n"
    "v_accvgpr_write_b32 a134, 0\n"
    "v_accvgpr_write_b32 a135, 0\n"
    "v_accvgpr_write_b32 a136, 0\n"
    "v_accvgpr_write_b32 a137, 0\n"
    "v_accvgpr_write_b32 a138, 0\n"
    "v_accvgpr_write_b32 a139, 0\n"
    "v_accvgpr_write_b32 a140, 0\n"
    "v_accvgpr_write_b32 a141, 0\n"
    "v_accvgpr_write_b32 a142, 0\n"
    "v_accvgpr_write_b32 a143, 0\n"
    "v_accvgpr_write_b32 a144, 0\n"
    "v_accvgpr_write_b32 a145, 0\n"
    "v_accvgpr_write_b32 a146, 0\n"
    "v_accvgpr_write_b32 a147, 0\n"
    "v_accvgpr_write_b32 a148, 0\n"
    "v_accvgpr_write_b32 a149, 0\n"
    "v_accvgpr_write_b32 a150, 0\n"
    "v_accvgpr_write_b32 a151, 0\n"
    "v_accvgpr_write_b32 a152, 0\n"
    "v_accvgpr_write_b32 a153, 0\n"
    "v_accvgpr_write_b32 a154, 0\n"
    "v_accvgpr_write_b32 a155, 0\n"
    "v_accvgpr_write_b32 a156, 0\n"
    "v_accvgpr_write_b32 a157, 0\n"
    "v_accvgpr_write_b32 a158, 0\n"
    "v_accvgpr_write_b32 a159, 0\n"
    "v_accvgpr_write_b32 a160, 0\n"
    "v_accvgpr_write_b32 a161, 0\n"
    "v_accvgpr_write_b32 a162, 0\n"
    "v_accvgpr_write_b32 a163, 0\n"
    "v_accvgpr_write_b32 a164, 0\n"
    "v_accvgpr_write_b32 a165, 0\n"
    "v_accvgpr_write_b32 a166, 0\n"
    "v_accvgpr_write_b32 a167, 0\n"
    "v_accvgpr_write_b32 a168, 0\n"
    "v_accvgpr_write_b32 a169, 0\n"
    "v_accvgpr_write_b32 a170, 0\n"
    "v_accvgpr_write_b32 a171, 0\n"
    "v_accvgpr_write_b32 a172, 0\n"
    "v_accvgpr_write_b32 a173, 0\n"
    "v_accvgpr_write_b32 a174, 0\n"
    "v_accvgpr_write_b32 a175, 0\n"
    "v_accvgpr_write_b32 a176, 0\n"
    "v_accvgpr_write_b32 a177, 0\n"
    "v_accvgpr_write_b32 a178, 0\n"
    "v_accvgpr_write_b32 a179, 0\n"
    "v_accvgpr_write_b32 a180, 0\n"
    "v_accvgpr_write_b32 a181, 0\n"
    "v_accvgpr_write_b32 a182, 0\n"
    "v_accvgpr_write_b32 a183, 0\n"
    "v_accvgpr_write_b32 a184, 0\n"
    "v_accvgpr_write_b32 a185, 0\n"
    "v_accvgpr_write_b32 a186, 0\n"
    "v_accvgpr_write_b32 a187, 0\n"
    "v_accvgpr_write_b32 a188, 0\n"
    "v_accvgpr_write_b32 a189, 0\n"
    "v_accvgpr_write_b32 a190, 0\n"
    "v_accvgpr_write_b32 a191, 0\n"
    "v_accvgpr_write_b32 a192, 0\n"
    "v_accvgpr_write_b32 a193, 0\n"
    "v_accvgpr_write_b32 a194, 0\n"
    "v_accvgpr_write_b32 a195, 0\n"
    "v_accvgpr_write_b32 a196, 0\n"
    "v_accvgpr_write_b32 a197, 0\n"
    "v_accvgpr_write_b32 a198, 0\n"
    "v_accvgpr_write_b32 a199, 0\n"
    "v_accvgpr_write_b32 a200, 0\n"
    "v_accvgpr_write_b32 a201, 0\n"
    "v_accvgpr_write_b32 a202, 0\n"
    "v_accvgpr_write_b32 a203, 0\n"
    "v_accvgpr_write_b32 a204, 0\n"
    "v_accvgpr_write_b32 a205, 0\n"
    "v_accvgpr_write_b32 a206, 0\n"
    "v_accvgpr_write_b32 a207, 0\n"
    "v_accvgpr_write_b32 a208, 0\n"
    "v_accvgpr_write_b32 a209, 0\n"
    "v_accvgpr_write_b32 a210, 0\n"
    "v_accvgpr_write_b32 a211, 0\n"
    "v_accvgpr_write_b32 a212, 0\n"
    "v_accvgpr_write_b32 a213, 0\n"
    "v_accvgpr_write_b32 a214, 0\n"
    "v_accvgpr_write_b32 a215, 0\n"
    "v_accvgpr_write_b32 a216, 0\n"
    "v_accvgpr_write_b32 a217, 0\n"
    "v_accvgpr_write_b32 a218, 0\n"
    "v_accvgpr_write_b32 a219, 0\n"
    "v_accvgpr_write_b32 a220, 0\n"
    "v_accvgpr_write_b32 a221, 0\n"
    "v_accvgpr_write_b32 a222, 0\n"
    "v_accvgpr_write_b32 a223, 0\n"
    "v_accvgpr_write_b32 a224, 0\n"
    "v_accvgpr_write_b32 a225, 0\n"
    "v_accvgpr_write_b32 a226, 0\n"
    "v_accvgpr_write_b32 a227, 0\n"
    "v_accvgpr_write_b32 a228, 0\n"
    "v_accvgpr_write_b32 a229, 0\n"
    "v_accvgpr_write_b32 a230, 0\n"
    "v_accvgpr_write_b32 a231, 0\n"
    "v_accvgpr_write_b32 a232, 0\n"
    "v_accvgpr_write_b32 a233, 0\n"
    "v_accvgpr_write_b32 a234, 0\n"
    "v_accvgpr_write_b32 a235, 0\n"
    "v_accvgpr_write_b32 a236, 0\n"
    "v_accvgpr_write_b32 a237, 0\n"
    "v_accvgpr_write_b32 a238, 0\n"
    "v_accvgpr_write_b32 a239, 0\n"
    "v_accvgpr_write_b32 a240, 0\n"
    "v_accvgpr_write_b32 a241, 0\n"
    "v_accvgpr_write_b32 a242, 0\n"
    "v_accvgpr_write_b32 a243, 0\n"
    "v_accvgpr_write_b32 a244, 0\n"
    "v_accvgpr_write_b32 a245, 0\n"
    "v_accvgpr_write_b32 a246, 0\n"
    "v_accvgpr_write_b32 a247, 0\n"
    "v_accvgpr_write_b32 a248, 0\n"
    "v_accvgpr_write_b32 a249, 0\n"
    "v_accvgpr_write_b32 a250, 0\n"
    "v_accvgpr_write_b32 a251, 0\n"
    "v_accvgpr_write_b32 a252, 0\n"
    "v_accvgpr_write_b32 a253, 0\n"
    "v_accvgpr_write_b32 a254, 0\n"
    "v_accvgpr_write_b32 a255, 0\n"
    "v_sub_u32 v23, v6, v10\n"
    "s_waitcnt vmcnt(8)\n"
    "s_cmp_eq_u32 s51, 0\n"
    "s_cbranch_scc0 NOPOLL0%=\n"
    "buffer_load_dwordx2 v[20:21], v13, s[72:75], 0 offen sc0 sc1\n"
    "NOPOLL0%=:\n"
    "v_cvt_pk_f16_f32 v40, v192, v196\n"
    "v_cvt_pk_f16_f32 v41, v200, v204\n"
    "v_cvt_pk_f16_f32 v42, v208, v212\n"
    "v_cvt_pk_f16_f32 v43, v216, v220\n"
    "ds_write_b128 v7, v[40:43] offset:0\n"
    "v_cvt_pk_f16_f32 v44, v193, v197\n"
    "v_cvt_pk_f16_f32 v45, v201, v205\n"
    "v_cvt_pk_f16_f32 v46, v209, v213\n"
    "v_cvt_pk_f16_f32 v47, v217, v221\n"
    "ds_write_b128 v7, v[44:47] offset:1024\n"
    "v_cvt_pk_f16_f32 v40, v194, v198\n"
    "v_cvt_pk_f16_f32 v41, v202, v206\n"
    "v_cvt_pk_f16_f32 v42, v210, v214\n"
    "v_cvt_pk_f16_f32 v43, v218, v222\n"
    "ds_write_b128 v7, v[40:43] offset:2048\n"
    "v_cvt_pk_f16_f32 v44, v195, v199\n"
    "v_cvt_pk_f16_f32 v45, v203, v207\n"
    "v_cvt_pk_f16_f32 v46, v211, v215\n"
    "v_cvt_pk_f16_f32 v47, v219, v223\n"
    "ds_write_b128 v7, v[44:47] offset:3072\n"
    "s_lshl_b32 s49, s52, 15\n"
    "s_add_u32 s50, s49, 4096\n"
    "s_cmp_eq_u32 s51, 0\n"
    "s_cbranch_scc0 NOTW0%=\n"
    "s_waitcnt vmcnt(0)\n"
    "v_cmp_gt_u32 vcc, v18, v20\n"
    "s_nop 4\n"
    "s_cbranch_vccnz SLOWP%=\n"
    "BACKP%=:\n"
    "v_sub_u32 v21, v21, v20\n"
    "s_nop 0\n"
    "v_mov_b32 v20, v21\n"
    "v_sub_u32 v22, 0, v21\n"
    "v_max_i32 v22, v22, v21\n"
    "s_nop 1\n"
    "v_add_u32_dpp v20, v20, v20 row_shr:1 bound_ctrl:0\n"
    "v_add_u32_dpp v22, v22, v22 row_shr:1 bound_ctrl:0\n"
    "s_nop 1\n"
    "v_add_u32_dpp v20, v20, v20 row_shr:2 bound_ctrl:0\n"
    "v_add_u32_dpp v22, v22, v22 row_shr:2 bound_ctrl:0\n"
    "s_nop 1\n"
    "v_add_u32_dpp v20, v20, v20 row_shr:4 bound_ctrl:0\n"
    "v_add_u32_dpp v22, v22, v22 row_shr:4 bound_ctrl:0\n"
    "s_nop 1\n"
    "v_add_u32_dpp v20, v20, v20 row_shr:8 bound_ctrl:0\n"
    "v_add_u32_dpp v22, v22, v22 row_shr:8 bound_ctrl:0\n"
    "s_nop 1\n"
    "v_readlane_b32 s80, v20, 31\n"
    "v_readlane_b32 s81, v22, 31\n"
    "s_nop 3\n"
    "s_abs_i32 s56, s80\n"
    "s_lshl_b32 s56, s56, 1\n"
    "s_cmp_ge_u32 s56, s81\n"
    "s_cselect_b32 s56, 5, 2\n"
    "s_sub_i32 s47, 0, s56\n"
    "v_cmp_le_i32 s[88:89], s56, v21\n"
    "v_cmp_ge_i32 vcc, s47, v21\n"
    "s_nop 3\n"
    "s_lshr_b32 s88, s88, 16\n"
    "s_lshr_b32 s89, vcc_lo, 16\n"
    "s_or_b32 s89, s88, s89\n"
    "s_cmp_gt_i32 s80, 0\n"
    "s_cselect_b32 s99, 0xffff, 0\n"
    "s_andn2_b32 s99, s99, s89\n"
    "s_or_b32 s88, s88, s99\n"
    "s_cmp_lt_u32 s81, 12\n"
    "s_cselect_b32 s80, 0x100000, 0\n"
    "s_or_b32 s80, s80, s88\n"
    "v_readlane_b32 s88, v21, 32\n"
    "v_readlane_b32 s56, v21, 33\n"
    "v_readlane_b32 s89, v21, 34\n"
    "v_readlane_b32 s47, v21, 35\n"
    "s_nop 3\n"
    "s_add_i32 s88, s88, s56\n"
    "s_add_i32 s89, s89, s47\n"
    "s_add_i32 s47, s88, s89\n"
    "s_abs_i32 s56, s88\n"
    "s_cmp_lt_i32 s56, 4\n"
    "s_cselect_b32 s88, s47, s88\n"
    "s_abs_i32 s56, s88\n"
    "s_cmp_gt_i32 s88, 0\n"
    "s_cselect_b32 s88, 1, 0\n"
    "s_cmp_lt_i32 s56, 4\n"
    "s_cselect_b32 s88, 2, s88\n"
    "s_lshl_b32 s88, s88, 16\n"
    "s_or_b32 s80, s80, s88\n"
    "s_abs_i32 s56, s89\n"
    "s_cmp_lt_i32 s56, 4\n"
    "s_cselect_b32 s89, s47, s89\n"
    "s_abs_i32 s56, s89\n"
    "s_cmp_gt_i32 s89, 0\n"
    "s_cselect_b32 s89, 1, 0\n"
    "s_cmp_lt_i32 s56, 4\n"
    "s_cselect_b32 s89, 2, s89\n"
    "s_lshl_b32 s89, s89, 18\n"
    "s_or_b32 s80, s80, s89\n"
    "v_mov_b32 v20, s80\n"
    "ds_write_b32 v23, v20 offset:32768\n"
    "NOTW0%=:\n"
    "s_waitcnt lgkmcnt(0)\n"
    "s_barrier\n"
    "ds_read_b32 v20, v23 offset:32768\n"
    "v_bfe_u32 v21, v10, 4, 1\n"
    "s_waitcnt lgkmcnt(0)\n"
    "v_readfirstlane_b32 s80, v20\n"
    "v_lshrrev_b32 v22, 3, v2\n"
    "v_cmp_lt_u32 vcc, 0xfffff, v20\n"
    "v_lshrrev_b32 v20, v22, v20\n"
    "s_nop 0\n"
    "v_cndmask_b32 v20, v20, v21, vcc\n"
    "v_and_b32 v20, v20, v14\n"
    "v_xor_b32 v2, v2, v20\n"
    "s_and_b32 s81, s51, 2\n"
    "s_add_u32 s81, s81, 16\n"
    "s_lshr_b32 s80, s80, s81\n"
    "s_and_b32 s80, s80, 3\n"
    "s_and_b32 s81, s51, 1\n"
    "s_cmp_eq_u32 s80, 2\n"
    "s_cselect_b32 s80, s81, s80\n"
    "s_cmp_eq_u32 s80, 0\n"
    "s_cbranch_scc1 NOFLIP%=\n"
    "v_xor_b32 v4, 0x800, v4\n"
    "v_xor_b32 v5, 0x800, v5\n"
    "s_mov_b64 s[80:81], s[64:65]\n"
    "s_mov_b64 s[64:65], s[68:69]\n"
    "s_mov_b64 s[68:69], s[80:81]\n"
    "s_mov_b64 s[80:81], s[66:67]\n"
    "s_mov_b64 s[66:67], s[70:71]\n"
    "s_mov_b64 s[70:71], s[80:81]\n"
    "NOFLIP%=:\n"
    "s_add_u32 s55, s55, 2\n"
    "buffer_load_dwordx4 v[64:67], v4, s[40:43], s49 offen offset:0 sc0 sc1\n"
    "buffer_load_dwordx4 v[68:71], v4, s[40:43], s49 offen offset:1024 sc0 sc1\n"
    "buffer_load_dwordx4 v[72:75], v4, s[40:43], s50 offen offset:0 sc0 sc1\n"
    "buffer_load_dwordx4 v[76:79], v4, s[40:43], s50 offen offset:1024 sc0 sc1\n"
    "buffer_load_dwordx4 v[80:83], v5, s[40:43], s49 offen offset:0 sc0 sc1\n"
    "buffer_load_dwordx4 v[84:87], v5, s[40:43], s49 offen offset:1024 sc0 sc1\n"
    "buffer_load_dwordx4 v[88:91], v5, s[40:43], s50 offen offset:0 sc0 sc1\n"
    "buffer_load_dwordx4 v[92:95], v5, s[40:43], s50 offen offset:1024 sc0 sc1\n"
    "v_readlane_b32 s52, v2, 2\n"
    "s_lshl_b32 s49, s46, 15\n"
    "s_add_u32 s50, s49, 4096\n"
    "buffer_load_dwordx4 v[96:99], v4, s[40:43], s49 offen offset:0 sc0 sc1\n"
    "buffer_load_dwordx4 v[100:103], v4, s[40:43], s49 offen offset:1024 sc0 sc1\n"
    "buffer_load_dwordx4 v[104:107], v4, s[40:43], s50 offen offset:0 sc0 sc1\n"
    "buffer_load_dwordx4 v[108:111], v4, s[40:43], s50 offen offset:1024 sc0 sc1\n"
    "buffer_load_dwordx4 v[112:115], v5, s[40:43], s49 offen offset:0 sc0 sc1\n"
    "buffer_load_dwordx4 v[116:119], v5, s[40:43], s49 offen offset:1024 sc0 sc1\n"
    "buffer_load_dwordx4 v[120:123], v5, s[40:43], s50 offen offset:0 sc0 sc1\n"
    "buffer_load_dwordx4 v[124:127], v5, s[40:43], s50 offen offset:1024 sc0 sc1\n"
    "buffer_load_dwordx4 v[160:163], v9, s[60:63], s77 offen offset:0 nt\n"
    "buffer_load_dwordx4 v[164:167], v9, s[60:63], s77 offen offset:16 nt\n"
    "buffer_load_dwordx4 v[168:171], v9, s[60:63], s78 offen offset:0 nt\n"
    "buffer_load_dwordx4 v[172:175], v9, s[60:63], s78 offen offset:16 nt\n"
    "buffer_load_dwordx4 v[176:179], v9, s[60:63], s79 offen offset:0 nt\n"
    "buffer_load_dwordx4 v[180:183], v9, s[60:63], s79 offen offset:16 nt\n"
    "s_lshl_b32 s49, s52, 15\n"
    "s_add_u32 s50, s49, 4096\n"
    "s_mov_b32 s46, 3\n"
    "s_mov_b32 s54, 0\n"
    "v_readfirstlane_b32 s56, v25\n"
    "s_mov_b32 s34, 201\n"
    "s_mov_b32 s53, 3300\n"
    "s_mov_b32 s35, 171\n"
    "s_mov_b32 s33, 4300\n"
    "s_mov_b32 s31, 156\n"
    "s_sub_u32 s56, s30, s56\n"
    "s_cmp_lt_u32 s56, 20000\n"
    "s_cselect_b32 s34, 170, s34\n"
    "s_cselect_b32 s53, 3600, s53\n"
    "s_cselect_b32 s35, 160, s35\n"
    "s_cselect_b32 s33, 9000, s33\n"
    "s_cselect_b32 s31, 160, s31\n"
    "s_mov_b32 s76, s30\n"
    "ds_read_b128 v[48:51], v6 offset:0\n"
    "ds_read_b128 v[52:55], v6 offset:8192\n"
    "s_waitcnt vmcnt(6) lgkmcnt(0)\n"
    "v_mfma_f32_16x16x32_f16 a[0:3], v[64:67], v[48:51], a[0:3]\n"
    "ds_read_b128 v[56:59], v6 offset:1024\n"
    "v_mfma_f32_16x16x32_f16 a[0:3], v[96:99], v[52:55], a[0:3]\n"
    "v_mfma_f32_16x16x32_f16 a[32:35], v[68:71], v[48:51], a[32:35]\n"
    "buffer_load_dwordx4 v[128:131], v4, s[40:43], s49 offen offset:0 sc0 sc1\n"
    "v_mfma_f32_16x16x32_f16 a[32:35], v[100:103], v[52:55], a[32:35]\n"
    "v_mfma_f32_16x16x32_f16 a[64:67], v[72:75], v[48:51], a[64:67]\n"
    "ds_read_b128 v[60:63], v6 offset:9216\n"
    "v_mfma_f32_16x16x32_f16 a[64:67], v[104:107], v[52:55], a[64:67]\n"
    "v_mfma_f32_16x16x32_f16 a[96:99], v[76:79], v[48:51], a[96:99]\n"
    "buffer_load_dwordx4 v[132:135], v4, s[40:43], s49 offen offset:1024 sc0 sc1\n"
    "v_mfma_f32_16x16x32_f16 a[96:99], v[108:111], v[52:55], a[96:99]\n"
    "v_mfma_f32_16x16x32_f16 a[128:131], v[80:83], v[48:51], a[128:131]\n"
    "v_mfma_f32_16x16x32_f16 a[128:131], v[112:115], v[52:55], a[128:131]\n"
    "v_mfma_f32_16x16x32_f16 a[160:163], v[84:87], v[48:51], a[160:163]\n"
    "buffer_load_dwordx4 v[136:139], v4, s[40:43], s50 offen offset:0 sc0 sc1\n"
    "v_mfma_f32_16x16x32_f16 a[160:163], v[116:119], v[52:55], a[160:163]\n"
    "v_mfma_f32_16x16x32_f16 a[192:195], v[88:91], v[48:51], a[192:195]\n"
    "v_mfma_f32_16x16x32_f16 a[192:195], v[120:123], v[52:55], a[192:195]\n"
    "v_mfma_f32_16x16x32_f16 a[224:227], v[92:95], v[48:51], a[224:227]\n"
    "buffer_load_dwordx4 v[140:143], v4, s[40:43], s50 offen offset:1024 sc0 sc1\n"
    "v_mfma_f32_16x16x32_f16 a[224:227], v[124:127], v[52:55], a[224:227]\n"
    "s_waitcnt lgkmcnt(0)\n"
    "s_waitcnt vmcnt(4)\n"
    "v_mfma_f32_16x16x32_f16 a[228:231], v[92:95], v[56:59], a[228:231]\n"
    "ds_read_b128 v[48:51], v6 offset:2048\n"
    "v_cvt_pk_f16_f32 v28, v160, v161\n"
    "v_mfma_f32_16x16x32_f16 a[228:231], v[124:127], v[60:63], a[228:231]\n"
    "v_cvt_pk_f16_f32 v29, v162, v163\n"
    "v_mfma_f32_16x16x32_f16 a[196:199], v[88:91], v[56:59], a[196:199]\n"
    "buffer_load_dwordx4 v[144:147], v5, s[40:43], s49 offen offset:0 sc0 sc1\n"
    "v_mfma_f32_16x16x32_f16 a[196:199], v[120:123], v[60:63], a[196:199]\n"
    "v_cvt_pk_f16_f32 v30, v164, v165\n"
    "v_mfma_f32_16x16x32_f16 a[164:167], v[84:87], v[56:59], a[164:167]\n"
    "ds_read_b128 v[52:55], v6 offset:10240\n"
    "v_cvt_pk_f16_f32 v31, v166, v167\n"
    "v_mfma_f32_16x16x32_f16 a[164:167], v[116:119], v[60:63], a[164:167]\n"
    "v_cvt_pk_f16_f32 v32, v168, v169\n"
    "v_mfma_f32_16x16x32_f16 a[132:135], v[80:83], v[56:59], a[132:135]\n"
    "buffer_load_dwordx4 v[148:151], v5, s[40:43], s49 offen offset:1024 sc0 sc1\n"
    "v_mfma_f32_16x16x32_f16 a[132:135], v[112:115], v[60:63], a[132:135]\n"
    "v_cvt_pk_f16_f32 v33, v170, v171\n"
    "v_mfma_f32_16x16x32_f16 a[100:103], v[76:79], v[56:59], a[100:103]\n"
    "v_cvt_pk_f16_f32 v34, v172, v173\n"
    "v_mfma_f32_16x16x32_f16 a[100:103], v[108:111], v[60:63], a[100:103]\n"
    "v_cvt_pk_f16_f32 v35, v174, v175\n"
    "v_mfma_f32_16x16x32_f16 a[68:71], v[72:75], v[56:59], a[68:71]\n"
    "buffer_load_dwordx4 v[152:155], v5, s[40:43], s50 offen offset:0 sc0 sc1\n"
    "v_mfma_f32_16x16x32_f16 a[68:71], v[104:107], v[60:63], a[68:71]\n"
    "v_cvt_pk_f16_f32 v36, v176, v177\n"
    "v_mfma_f32_16x16x32_f16 a[36:39], v[68:71], v[56:59], a[36:39]\n"
    "v_cvt_pk_f16_f32 v37, v178, v179\n"
    "v_mfma_f32_16x16x32_f16 a[36:39], v[100:103], v[60:63], a[36:39]\n"
    "v_cvt_pk_f16_f32 v38, v180, v181\n"
    "v_mfma_f32_16x16x32_f16 a[4:7], v[64:67], v[56:59], a[4:7]\n"
    "buffer_load_dwordx4 v[156:159], v5, s[40:43], s50 offen offset:1024 sc0 sc1\n"
    "v_mfma_f32_16x16x32_f16 a[4:7], v[96:99], v[60:63], a[4:7]\n"
    "v_readlane_b32 s52, v2, s46\n"
    "v_cvt_pk_f16_f32 v39, v182, v183\n"
    "s_waitcnt lgkmcnt(0)\n"
    "v_mfma_f32_16x16x32_f16 a[8:11], v[64:67], v[48:51], a[8:11]\n"
    "ds_read_b128 v[56:59], v6 offset:3072\n"
    "s_cmp_lt_u32 s46, 64\n"
    "s_cselect_b32 s42, s45, 0\n"
    "v_mfma_f32_16x16x32_f16 a[8:11], v[96:99], v[52:55], a[8:11]\n"
    "s_lshl_b32 s49, s52, 15\n"
    "s_add_u32 s50, s49, 4096\n"
    "s_add_u32 s46, s46, 1\n"
    "v_mfma_f32_16x16x32_f16 a[40:43], v[68:71], v[48:51], a[40:43]\n"
    "buffer_load_dwordx4 v[160:163], v4, s[40:43], s49 offen offset:0 sc0 sc1\n"
    "v_mfma_f32_16x16x32_f16 a[40:43], v[100:103], v[52:55], a[40:43]\n"
    "s_cmp_eq_u32 s58, 0\n"
    "s_cbranch_scc0 NOPUSH%=\n"
    "global_store_dwordx4 v10, v[28:31], s[82:83] sc0 sc1\n"
    "NOPUSH%=:\n"
    "v_mfma_f32_16x16x32_f16 a[72:75], v[72:75], v[48:51], a[72:75]\n"
    "ds_read_b128 v[60:63], v6 offset:11264\n"
    "v_mfma_f32_16x16x32_f16 a[72:75], v[104:107], v[52:55], a[72:75]\n"
    "v_mfma_f32_16x16x32_f16 a[104:107], v[76:79], v[48:51], a[104:107]\n"
    "buffer_load_dwordx4 v[164:167], v4, s[40:43], s49 offen offset:1024 sc0 sc1\n"
    "v_mfma_f32_16x16x32_f16 a[104:107], v[108:111], v[52:55], a[104:107]\n"
    "v_mfma_f32_16x16x32_f16 a[136:139], v[80:83], v[48:51], a[136:139]\n"
    "v_mfma_f32_16x16x32_f16 a[136:139], v[112:115], v[52:55], a[136:139]\n"
    "v_mfma_f32_16x16x32_f16 a[168:171], v[84:87], v[48:51], a[168:171]\n"
    "buffer_load_dwordx4 v[168:171], v4, s[40:43], s50 offen offset:0 sc0 sc1\n"
    "v_mfma_f32_16x16x32_f16 a[168:171], v[116:119], v[52:55], a[168:171]\n"
    "v_mfma_f32_16x16x32_f16 a[200:203], v[88:91], v[48:51], a[200:203]\n"
    "v_mfma_f32_16x16x32_f16 a[200:203], v[120:123], v[52:55], a[200:203]\n"
    "v_cvt_pk_f16_f32 v40, v224, v228\n"
    "v_mfma_f32_16x16x32_f16 a[232:235], v[92:95], v[48:51], a[232:235]\n"
    "buffer_load_dwordx4 v[172:175], v4, s[40:43], s50 offen offset:1024 sc0 sc1\n"
    "v_mfma_f32_16x16x32_f16 a[232:235], v[124:127], v[52:55], a[232:235]\n"
    "s_waitcnt lgkmcnt(0)\n"
    "v_mfma_f32_16x16x32_f16 a[236:239], v[92:95], v[56:59], a[236:239]\n"
    "ds_read_b128 v[48:51], v6 offset:4096\n"
    "v_mfma_f32_16x16x32_f16 a[236:239], v[124:127], v[60:63], a[236:239]\n"
    "v_cvt_pk_f16_f32 v41, v232, v236\n"
    "v_mfma_f32_16x16x32_f16 a[204:207], v[88:91], v[56:59], a[204:207]\n"
    "buffer_load_dwordx4 v[176:179], v5, s[40:43], s49 offen offset:0 sc0 sc1\n"
    "v_mfma_f32_16x16x32_f16 a[204:207], v[120:123], v[60:63], a[204:207]\n"
    "v_mfma_f32_16x16x32_f16 a[172:175], v[84:87], v[56:59], a[172:175]\n"
    "ds_read_b128 v[52:55], v6 offset:12288\n"
    "v_mfma_f32_16x16x32_f16 a[172:175], v[116:119], v[60:63], a[172:175]\n"
    "v_cvt_pk_f16_f32 v42, v240, v244\n"
    "v_mfma_f32_16x16x32_f16 a[140:143], v[80:83], v[56:59], a[140:143]\n"
    "buffer_load_dwordx4 v[180:183], v5, s[40:43], s49 offen offset:1024 sc0 sc1\n"
    "v_mfma_f32_16x16x32_f16 a[140:143], v[112:115], v[60:63], a[140:143]\n"
    "v_readlane_b32 s47, v2, s55\n"
    "v_mfma_f32_16x16x32_f16 a[108:111], v[76:79], v[56:59], a[108:111]\n"
    "v_mfma_f32_16x16x32_f16 a[108:111], v[108:111], v[60:63], a[108:111]\n"
    "v_cvt_pk_f16_f32 v43, v248, v252\n"
    "v_mfma_f32_16x16x32_f16 a[76:79], v[72:75], v[56:59], a[76:79]\n"
    "buffer_load_dwordx4 v[184:187], v5, s[40:43], s50 offen offset:0 sc0 sc1\n"
    "v_mfma_f32_16x16x32_f16 a[76:79], v[104:107], v[60:63], a[76:79]\n"
    "s_cmp_lt_u32 s55, 64\n"
    "s_cselect_b32 s38, s44, 0\n"
    "v_mfma_f32_16x16x32_f16 a[44:47], v[68:71], v[56:59], a[44:47]\n"
    "ds_write_b128 v7, v[40:43] offset:16384\n"
    "v_mfma_f32_16x16x32_f16 a[44:47], v[100:103], v[60:63], a[44:47]\n"
    "v_cvt_pk_f16_f32 v44, v225, v229\n"
    "v_mfma_f32_16x16x32_f16 a[12:15], v[64:67], v[56:59], a[12:15]\n"
    "buffer_load_dwordx4 v[188:191], v5, s[40:43], s50 offen offset:1024 sc0 sc1\n"
    "v_mfma_f32_16x16x32_f16 a[12:15], v[96:99], v[60:63], a[12:15]\n"
    "v_readlane_b32 s52, v2, s46\n"
    "s_lshl_b32 s48, s47, 21\n"
    "s_add_u32 s55, s55, 2\n"
    "s_waitcnt lgkmcnt(0)\n"
    "v_mfma_f32_16x16x32_f16 a[16:19], v[64:67], v[48:51], a[16:19]\n"
    "ds_read_b128 v[56:59], v6 offset:5120\n"
    "s_cmp_lt_u32 s46, 64\n"
    "s_cselect_b32 s42, s45, 0\n"
    "v_mfma_f32_16x16x32_f16 a[16:19], v[96:99], v[52:55], a[16:19]\n"
    "s_lshl_b32 s49, s52, 15\n"
    "s_add_u32 s50, s49, 4096\n"
    "s_add_u32 s46, s46, 1\n"
    "v_cvt_pk_f16_f32 v45, v233, v237\n"
    "v_mfma_f32_16x16x32_f16 a[48:51], v[68:71], v[48:51], a[48:51]\n"
    "buffer_load_dwordx4 v[192:195], v3, s[36:39], s48 offen nt\n"
    "v_mfma_f32_16x16x32_f16 a[48:51], v[100:103], v[52:55], a[48:51]\n"
    "s_add_u32 s48, s48, 0x10000\n"
    "v_mfma_f32_16x16x32_f16 a[80:83], v[72:75], v[48:51], a[80:83]\n"
    "ds_read_b128 v[60:63], v6 offset:13312\n"
    "v_mfma_f32_16x16x32_f16 a[80:83], v[104:107], v[52:55], a[80:83]\n"
    "v_cvt_pk_f16_f32 v46, v241, v245\n"
    "v_mfma_f32_16x16x32_f16 a[112:115], v[76:79], v[48:51], a[112:115]\n"
    "buffer_load_dwordx4 v[196:199], v3, s[36:39], s48 offen nt\n"
    "v_mfma_f32_16x16x32_f16 a[112:115], v[108:111], v[52:55], a[112:115]\n"
    "s_add_u32 s48, s48, 0x10000\n"
    "v_mfma_f32_16x16x32_f16 a[144:147], v[80:83], v[48:51], a[144:147]\n"
    "v_mfma_f32_16x16x32_f16 a[144:147], v[112:115], v[52:55], a[144:147]\n"
    "v_cvt_pk_f16_f32 v47, v249, v253\n"
    "v_mfma_f32_16x16x32_f16 a[176:179], v[84:87], v[48:51], a[176:179]\n"
    "buffer_load_dwordx4 v[200:203], v3, s[36:39], s48 offen nt\n"
    "v_mfma_f32_16x16x32_f16 a[176:179], v[116:119], v[52:55], a[176:179]\n"
    "s_add_u32 s48, s48, 0x10000\n"
    "v_mfma_f32_16x16x32_f16 a[208:211], v[88:91], v[48:51], a[208:211]\n"
    "ds_write_b128 v7, v[44:47] offset:17408\n"
    "v_mfma_f32_16x16x32_f16 a[208:211], v[120:123], v[52:55], a[208:211]\n"
    "v_cvt_pk_f16_f32 v40, v226, v230\n"
    "v_mfma_f32_16x16x32_f16 a[240:243], v[92:95], v[48:51], a[240:243]\n"
    "buffer_load_dwordx4 v[204:207], v3, s[36:39], s48 offen nt\n"
    "v_mfma_f32_16x16x32_f16 a[240:243], v[124:127], v[52:55], a[240:243]\n"
    "s_add_u32 s48, s48, 0x10000\n"
    "s_waitcnt lgkmcnt(0)\n"
    "v_mfma_f32_16x16x32_f16 a[244:247], v[92:95], v[56:59], a[244:247]\n"
    "ds_read_b128 v[48:51], v6 offset:6144\n"
    "v_mfma_f32_16x16x32_f16 a[244:247], v[124:127], v[60:63], a[244:247]\n"
    "v_cvt_pk_f16_f32 v41, v234, v238\n"
    "v_mfma_f32_16x16x32_f16 a[212:215], v[88:91], v[56:59], a[212:215]\n"
    "buffer_load_dwordx4 v[208:211], v3, s[36:39], s48 offen nt\n"
    "v_mfma_f32_16x16x32_f16 a[212:215], v[120:123], v[60:63], a[212:215]\n"
    "s_add_u32 s48, s48, 0x10000\n"
    "v_mfma_f32_16x16x32_f16 a[180:183], v[84:87], v[56:59], a[180:183]\n"
    "ds_read_b128 v[52:55], v6 offset:14336\n"
    "v_mfma_f32_16x16x32_f16 a[180:183], v[116:119], v[60:63], a[180:183]\n"
    "v_cvt_pk_f16_f32 v42, v242, v246\n"
    "v_mfma_f32_16x16x32_f16 a[148:151], v[80:83], v[56:59], a[148:151]\n"
    "buffer_load_dwordx4 v[212:215], v3, s[36:39], s48 offen nt\n"
    "v_mfma_f32_16x16x32_f16 a[148:151], v[112:115], v[60:63], a[148:151]\n"
    "s_add_u32 s48, s48, 0x10000\n"
    "v_mfma_f32_16x16x32_f16 a[116:119], v[76:79], v[56:59], a[116:119]\n"
    "v_mfma_f32_16x16x32_f16 a[116:119], v[108:111], v[60:63], a[116:119]\n"
    "v_cvt_pk_f16_f32 v43, v250, v254\n"
    "v_mfma_f32_16x16x32_f16 a[84:87], v[72:75], v[56:59], a[84:87]\n"
    "buffer_load_dwordx4 v[216:219], v3, s[36:39], s48 offen nt\n"
    "v_mfma_f32_16x16x32_f16 a[84:87], v[104:107], v[60:63], a[84:87]\n"
    "s_add_u32 s48, s48, 0x10000\n"
    "v_mfma_f32_16x16x32_f16 a[52:55], v[68:71], v[56:59], a[52:55]\n"
    "ds_write_b128 v7, v[40:43] offset:18432\n"
    "v_mfma_f32_16x16x32_f16 a[52:55], v[100:103], v[60:63], a[52:55]\n"
    "v_cvt_pk_f16_f32 v44, v227, v231\n"
    "v_mfma_f32_16x16x32_f16 a[20:23], v[64:67], v[56:59], a[20:23]\n"
    "buffer_load_dwordx4 v[220:223], v3, s[36:39], s48 offen nt\n"
    "v_mfma_f32_16x16x32_f16 a[20:23], v[96:99], v[60:63], a[20:23]\n"
    "s_waitcnt lgkmcnt(0)\n"
    "v_mfma_f32_16x16x32_f16 a[24:27], v[64:67], v[48:51], a[24:27]\n"
    "ds_read_b128 v[56:59], v6 offset:7168\n"
    "v_mfma_f32_16x16x32_f16 a[24:27], v[96:99], v[52:55], a[24:27]\n"
    "v_cvt_pk_f16_f32 v45, v235, v239\n"
    "v_mfma_f32_16x16x32_f16 a[56:59], v[68:71], v[48:51], a[56:59]\n"
    "v_mfma_f32_16x16x32_f16 a[56:59], v[100:103], v[52:55], a[56:59]\n"
    "v_mfma_f32_16x16x32_f16 a[88:91], v[72:75], v[48:51], a[88:91]\n"
    "ds_read_b128 v[60:63], v6 offset:15360\n"
    "v_mfma_f32_16x16x32_f16 a[88:91], v[104:107], v[52:55], a[88:91]\n"
    "v_cvt_pk_f16_f32 v46, v243, v247\n"
    "v_mfma_f32_16x16x32_f16 a[120:123], v[76:79], v[48:51], a[120:123]\n"
    "v_mfma_f32_16x16x32_f16 a[120:123], v[108:111], v[52:55], a[120:123]\n"
    "v_mfma_f32_16x16x32_f16 a[152:155], v[80:83], v[48:51], a[152:155]\n"
    "v_mfma_f32_16x16x32_f16 a[152:155], v[112:115], v[52:55], a[152:155]\n"
    "v_cvt_pk_f16_f32 v47, v251, v255\n"
    "v_mfma_f32_16x16x32_f16 a[184:187], v[84:87], v[48:51], a[184:187]\n"
    "v_mfma_f32_16x16x32_f16 a[184:187], v[116:119], v[52:55], a[184:187]\n"
    "v_mfma_f32_16x16x32_f16 a[216:219], v[88:91], v[48:51], a[216:219]\n"
    "ds_write_b128 v7, v[44:47] offset:19456\n"
    "v_mfma_f32_16x16x32_f16 a[216:219], v[120:123], v[52:55], a[216:219]\n"
    "v_mfma_f32_16x16x32_f16 a[248:251], v[92:95], v[48:51], a[248:251]\n"
    "v_mfma_f32_16x16x32_f16 a[248:251], v[124:127], v[52:55], a[248:251]\n"
    "s_memrealtime s[78:79]\n"
    "s_waitcnt lgkmcnt(0)\n"
    "s_sub_u32 s56, s78, s76\n"
    "s_cmp_lt_i32 s56, 0\n"
    "s_cbranch_scc1 PACEH%=\n"
    "PACEDH%=:\n"
    "s_barrier\n"
    "v_mfma_f32_16x16x32_f16 a[252:255], v[92:95], v[56:59], a[252:255]\n"
    "ds_read_b128 v[48:51], v6 offset:16384\n"
    "v_mfma_f32_16x16x32_f16 a[252:255], v[124:127], v[60:63], a[252:255]\n"
    "s_sub_u32 s79, s78, s30\n"
    "s_cmp_lt_u32 s79, s33\n"
    "s_cselect_b32 s56, s35, s31\n"
    "v_mfma_f32_16x16x32_f16 a[220:223], v[88:91], v[56:59], a[220:223]\n"
    "s_cmp_lt_u32 s79, s53\n"
    "s_cselect_b32 s56, s34, s56\n"
    "v_mfma_f32_16x16x32_f16 a[220:223], v[120:123], v[60:63], a[220:223]\n"
    "v_mfma_f32_16x16x32_f16 a[188:191], v[84:87], v[56:59], a[188:191]\n"
    "ds_read_b128 v[52:55], v6 offset:24576\n"
    "v_mfma_f32_16x16x32_f16 a[188:191], v[116:119], v[60:63], a[188:191]\n"
    "v_mfma_f32_16x16x32_f16 a[156:159], v[80:83], v[56:59], a[156:159]\n"
    "v_mfma_f32_16x16x32_f16 a[156:159], v[112:115], v[60:63], a[156:159]\n"
    "v_mfma_f32_16x16x32_f16 a[124:127], v[76:79], v[56:59], a[124:127]\n"
    "v_mfma_f32_16x16x32_f16 a[124:127], v[108:111], v[60:63], a[124:127]\n"
    "v_mfma_f32_16x16x32_f16 a[92:95], v[72:75], v[56:59], a[92:95]\n"
    "v_mfma_f32_16x16x32_f16 a[92:95], v[104:107], v[60:63], a[92:95]\n"
    "v_mfma_f32_16x16x32_f16 a[60:63], v[68:71], v[56:59], a[60:63]\n"
    "v_mfma_f32_16x16x32_f16 a[60:63], v[100:103], v[60:63], a[60:63]\n"
    "v_mfma_f32_16x16x32_f16 a[28:31], v[64:67], v[56:59], a[28:31]\n"
    "v_mfma_f32_16x16x32_f16 a[28:31], v[96:99], v[60:63], a[28:31]\n"
    "s_branch ENTRY%=\n"
    "LOOP%=:\n"
    "s_waitcnt vmcnt(8) lgkmcnt(0)\n"
    "v_mfma_f32_16x16x32_f16 a[0:3], v[64:67], v[48:51], a[0:3]\n"
    "ds_read_b128 v[56:59], v6 offset:1024\n"
    "v_mfma_f32_16x16x32_f16 a[0:3], v[96:99], v[52:55], a[0:3]\n"
    "v_mfma_f32_16x16x32_f16 a[32:35], v[68:71], v[48:51], a[32:35]\n"
    "buffer_load_dwordx4 v[128:131], v4, s[40:43], s49 offen offset:0 sc0 sc1\n"
    "v_mfma_f32_16x16x32_f16 a[32:35], v[100:103], v[52:55], a[32:35]\n"
    "v_mfma_f32_16x16x32_f16 a[64:67], v[72:75], v[48:51], a[64:67]\n"
    "ds_read_b128 v[60:63], v6 offset:9216\n"
    "v_mfma_f32_16x16x32_f16 a[64:67], v[104:107], v[52:55], a[64:67]\n"
    "v_mfma_f32_16x16x32_f16 a[96:99], v[76:79], v[48:51], a[96:99]\n"
    "buffer_load_dwordx4 v[132:135], v4, s[40:43], s49 offen offset:1024 sc0 sc1\n"
    "v_mfma_f32_16x16x32_f16 a[96:99], v[108:111], v[52:55], a[96:99]\n"
    "v_mfma_f32_16x16x32_f16 a[128:131], v[80:83], v[48:51], a[128:131]\n"
    "v_mfma_f32_16x16x32_f16 a[128:131], v[112:115], v[52:55], a[128:131]\n"
    "v_mfma_f32_16x16x32_f16 a[160:163], v[84:87], v[48:51], a[160:163]\n"
    "buffer_load_dwordx4 v[136:139], v4, s[40:43], s50 offen offset:0 sc0 sc1\n"
    "v_mfma_f32_16x16x32_f16 a[160:163], v[116:119], v[52:55], a[160:163]\n"
    "v_mfma_f32_16x16x32_f16 a[192:195], v[88:91], v[48:51], a[192:195]\n"
    "v_mfma_f32_16x16x32_f16 a[192:195], v[120:123], v[52:55], a[192:195]\n"
    "v_mfma_f32_16x16x32_f16 a[224:227], v[92:95], v[48:51], a[224:227]\n"
    "buffer_load_dwordx4 v[140:143], v4, s[40:43], s50 offen offset:1024 sc0 sc1\n"
    "v_mfma_f32_16x16x32_f16 a[224:227], v[124:127], v[52:55], a[224:227]\n"
    "s_waitcnt lgkmcnt(0)\n"
    "v_mfma_f32_16x16x32_f16 a[228:231], v[92:95], v[56:59], a[228:231]\n"
    "ds_read_b128 v[48:51], v6 offset:2048\n"
    "v_mfma_f32_16x16x32_f16 a[228:231], v[124:127], v[60:63], a[228:231]\n"
    "v_mfma_f32_16x16x32_f16 a[196:199], v[88:91], v[56:59], a[196:199]\n"
    "buffer_load_dwordx4 v[144:147], v5, s[40:43], s49 offen offset:0 sc0 sc1\n"
    "v_mfma_f32_16x16x32_f16 a[196:199], v[120:123], v[60:63], a[196:199]\n"
    "v_mfma_f32_16x16x32_f16 a[164:167], v[84:87], v[56:59], a[164:167]\n"
    "ds_read_b128 v[52:55], v6 offset:10240\n"
    "v_mfma_f32_16x16x32_f16 a[164:167], v[116:119], v[60:63], a[164:167]\n"
    "v_mfma_f32_16x16x32_f16 a[132:135], v[80:83], v[56:59], a[132:135]\n"
    "buffer_load_dwordx4 v[148:151], v5, s[40:43], s49 offen offset:1024 sc0 sc1\n"
    "v_mfma_f32_16x16x32_f16 a[132:135], v[112:115], v[60:63], a[132:135]\n"
    "v_mfma_f32_16x16x32_f16 a[100:103], v[76:79], v[56:59], a[100:103]\n"
    "v_mfma_f32_16x16x32_f16 a[100:103], v[108:111], v[60:63], a[100:103]\n"
    "v_mfma_f32_16x16x32_f16 a[68:71], v[72:75], v[56:59], a[68:71]\n"
    "buffer_load_dwordx4 v[152:155], v5, s[40:43], s50 offen offset:0 sc0 sc1\n"
    "v_mfma_f32_16x16x32_f16 a[68:71], v[104:107], v[60:63], a[68:71]\n"
    "v_mfma_f32_16x16x32_f16 a[36:39], v[68:71], v[56:59], a[36:39]\n"
    "v_mfma_f32_16x16x32_f16 a[36:39], v[100:103], v[60:63], a[36:39]\n"
    "v_mfma_f32_16x16x32_f16 a[4:7], v[64:67], v[56:59], a[4:7]\n"
    "buffer_load_dwordx4 v[156:159], v5, s[40:43], s50 offen offset:1024 sc0 sc1\n"
    "v_mfma_f32_16x16x32_f16 a[4:7], v[96:99], v[60:63], a[4:7]\n"
    "v_readlane_b32 s52, v2, s46\n"
    "s_waitcnt lgkmcnt(0)\n"
    "v_mfma_f32_16x16x32_f16 a[8:11], v[64:67], v[48:51], a[8:11]\n"
    "ds_read_b128 v[56:59], v6 offset:3072\n"
    "s_cmp_lt_u32 s46, 64\n"
    "s_cselect_b32 s42, s45, 0\n"
    "v_mfma_f32_16x16x32_f16 a[8:11], v[96:99], v[52:55], a[8:11]\n"
    "s_lshl_b32 s49, s52, 15\n"
    "s_add_u32 s50, s49, 4096\n"
    "s_add_u32 s46, s46, 1\n"
    "v_mfma_f32_16x16x32_f16 a[40:43], v[68:71], v[48:51], a[40:43]\n"
    "buffer_load_dwordx4 v[160:163], v4, s[40:43], s49 offen offset:0 sc0 sc1\n"
    "v_mfma_f32_16x16x32_f16 a[40:43], v[100:103], v[52:55], a[40:43]\n"
    "v_mfma_f32_16x16x32_f16 a[72:75], v[72:75], v[48:51], a[72:75]\n"
    "ds_read_b128 v[60:63], v6 offset:11264\n"
    "v_mfma_f32_16x16x32_f16 a[72:75], v[104:107], v[52:55], a[72:75]\n"
    "v_mfma_f32_16x16x32_f16 a[104:107], v[76:79], v[48:51], a[104:107]\n"
    "buffer_load_dwordx4 v[164:167], v4, s[40:43], s49 offen offset:1024 sc0 sc1\n"
    "v_mfma_f32_16x16x32_f16 a[104:107], v[108:111], v[52:55], a[104:107]\n"
    "v_mfma_f32_16x16x32_f16 a[136:139], v[80:83], v[48:51], a[136:139]\n"
    "v_mfma_f32_16x16x32_f16 a[136:139], v[112:115], v[52:55], a[136:139]\n"
    "v_mfma_f32_16x16x32_f16 a[168:171], v[84:87], v[48:51], a[168:171]\n"
    "buffer_load_dwordx4 v[168:171], v4, s[40:43], s50 offen offset:0 sc0 sc1\n"
    "v_mfma_f32_16x16x32_f16 a[168:171], v[116:119], v[52:55], a[168:171]\n"
    "s_waitcnt vmcnt(11)\n"
    "v_mfma_f32_16x16x32_f16 a[200:203], v[88:91], v[48:51], a[200:203]\n"
    "v_mfma_f32_16x16x32_f16 a[200:203], v[120:123], v[52:55], a[200:203]\n"
    "v_cvt_pk_f16_f32 v40, v224, v228\n"
    "v_mfma_f32_16x16x32_f16 a[232:235], v[92:95], v[48:51], a[232:235]\n"
    "buffer_load_dwordx4 v[172:175], v4, s[40:43], s50 offen offset:1024 sc0 sc1\n"
    "v_mfma_f32_16x16x32_f16 a[232:235], v[124:127], v[52:55], a[232:235]\n"
    "s_sub_u32 s56, s54, s58\n"
    "s_cmp_lt_u32 s56, 24\n"
    "s_cbranch_scc1 HOOK%=\n"
    "HBACK%=:\n"
    "s_waitcnt lgkmcnt(0)\n"
    "v_mfma_f32_16x16x32_f16 a[236:239], v[92:95], v[56:59], a[236:239]\n"
    "ds_read_b128 v[48:51], v6 offset:4096\n"
    "v_mfma_f32_16x16x32_f16 a[236:239], v[124:127], v[60:63], a[236:239]\n"
    "v_cvt_pk_f16_f32 v41, v232, v236\n"
    "v_mfma_f32_16x16x32_f16 a[204:207], v[88:91], v[56:59], a[204:207]\n"
    "buffer_load_dwordx4 v[176:179], v5, s[40:43], s49 offen offset:0 sc0 sc1\n"
    "v_mfma_f32_16x16x32_f16 a[204:207], v[120:123], v[60:63], a[204:207]\n"
    "v_mfma_f32_16x16x32_f16 a[172:175], v[84:87], v[56:59], a[172:175]\n"
    "ds_read_b128 v[52:55], v6 offset:12288\n"
    "v_mfma_f32_16x16x32_f16 a[172:175], v[116:119], v[60:63], a[172:175]\n"
    "v_cvt_pk_f16_f32 v42, v240, v244\n"
    "v_mfma_f32_16x16x32_f16 a[140:143], v[80:83], v[56:59], a[140:143]\n"
    "buffer_load_dwordx4 v[180:183], v5, s[40:43], s49 offen offset:1024 sc0 sc1\n"
    "v_mfma_f32_16x16x32_f16 a[140:143], v[112:115], v[60:63], a[140:143]\n"
    "v_readlane_b32 s47, v2, s55\n"
    "v_mfma_f32_16x16x32_f16 a[108:111], v[76:79], v[56:59], a[108:111]\n"
    "v_mfma_f32_16x16x32_f16 a[108:111], v[108:111], v[60:63], a[108:111]\n"
    "v_cvt_pk_f16_f32 v43, v248, v252\n"
    "v_mfma_f32_16x16x32_f16 a[76:79], v[72:75], v[56:59], a[76:79]\n"
    "buffer_load_dwordx4 v[184:187], v5, s[40:43], s50 offen offset:0 sc0 sc1\n"
    "v_mfma_f32_16x16x32_f16 a[76:79], v[104:107], v[60:63], a[76:79]\n"
    "s_cmp_lt_u32 s55, 64\n"
    "s_cselect_b32 s38, s44, 0\n"
    "v_mfma_f32_16x16x32_f16 a[44:47], v[68:71], v[56:59], a[44:47]\n"
    "ds_write_b128 v7, v[40:43] offset:16384\n"
    "v_mfma_f32_16x16x32_f16 a[44:47], v[100:103], v[60:63], a[44:47]\n"
    "v_cvt_pk_f16_f32 v44, v225, v229\n"
    "v_mfma_f32_16x16x32_f16 a[12:15], v[64:67], v[56:59], a[12:15]\n"
    "buffer_load_dwordx4 v[188:191], v5, s[40:43], s50 offen offset:1024 sc0 sc1\n"
    "v_mfma_f32_16x16x32_f16 a[12:15], v[96:99], v[60:63], a[12:15]\n"
    "v_readlane_b32 s52, v2, s46\n"
    "s_lshl_b32 s48, s47, 21\n"
    "s_add_u32 s55, s55, 2\n"
    "s_cmp_eq_u32 s51, 0\n"
    "s_cbranch_scc1 DOPOLL%=\n"
    "POLLED%=:\n"
    "s_waitcnt lgkmcnt(0)\n"
    "v_mfma_f32_16x16x32_f16 a[16:19], v[64:67], v[48:51], a[16:19]\n"
    "ds_read_b128 v[56:59], v6 offset:5120\n"
    "s_cmp_lt_u32 s46, 64\n"
    "s_cselect_b32 s42, s45, 0\n"
    "v_mfma_f32_16x16x32_f16 a[16:19], v[96:99], v[52:55], a[16:19]\n"
    "s_lshl_b32 s49, s52, 15\n"
    "s_add_u32 s50, s49, 4096\n"
    "s_add_u32 s46, s46, 1\n"
    "v_cvt_pk_f16_f32 v45, v233, v237\n"
    "v_mfma_f32_16x16x32_f16 a[48:51], v[68:71], v[48:51], a[48:51]\n"
    "buffer_load_dwordx4 v[192:195], v3, s[36:39], s48 offen nt\n"
    "v_mfma_f32_16x16x32_f16 a[48:51], v[100:103], v[52:55], a[48:51]\n"
    "s_add_u32 s48, s48, 0x10000\n"
    "v_mfma_f32_16x16x32_f16 a[80:83], v[72:75], v[48:51], a[80:83]\n"
    "ds_read_b128 v[60:63], v6 offset:13312\n"
    "v_mfma_f32_16x16x32_f16 a[80:83], v[104:107], v[52:55], a[80:83]\n"
    "v_cvt_pk_f16_f32 v46, v241, v245\n"
    "v_mfma_f32_16x16x32_f16 a[112:115], v[76:79], v[48:51], a[112:115]\n"
    "buffer_load_dwordx4 v[196:199], v3, s[36:39], s48 offen nt\n"
    "v_mfma_f32_16x16x32_f16 a[112:115], v[108:111], v[52:55], a[112:115]\n"
    "s_add_u32 s48, s48, 0x10000\n"
    "v_mfma_f32_16x16x32_f16 a[144:147], v[80:83], v[48:51], a[144:147]\n"
    "v_mfma_f32_16x16x32_f16 a[144:147], v[112:115], v[52:55], a[144:147]\n"
    "v_cvt_pk_f16_f32 v47, v249, v253\n"
    "v_mfma_f32_16x16x32_f16 a[176:179], v[84:87], v[48:51], a[176:179]\n"
    "buffer_load_dwordx4 v[200:203], v3, s[36:39], s48 offen nt\n"
    "v_mfma_f32_16x16x32_f16 a[176:179], v[116:119], v[52:55], a[176:179]\n"
    "s_add_u32 s48, s48, 0x10000\n"
    "v_mfma_f32_16x16x32_f16 a[208:211], v[88:91], v[48:51], a[208:211]\n"
    "ds_write_b128 v7, v[44:47] offset:17408\n"
    "v_mfma_f32_16x16x32_f16 a[208:211], v[120:123], v[52:55], a[208:211]\n"
    "v_cvt_pk_f16_f32 v40, v226, v230\n"
    "v_mfma_f32_16x16x32_f16 a[240:243], v[92:95], v[48:51], a[240:243]\n"
    "buffer_load_dwordx4 v[204:207], v3, s[36:39], s48 offen nt\n"
    "v_mfma_f32_16x16x32_f16 a[240:243], v[124:127], v[52:55], a[240:243]\n"
    "s_add_u32 s48, s48, 0x10000\n"
    "s_waitcnt lgkmcnt(0)\n"
    "v_mfma_f32_16x16x32_f16 a[244:247], v[92:95], v[56:59], a[244:247]\n"
    "ds_read_b128 v[48:51], v6 offset:6144\n"
    "v_mfma_f32_16x16x32_f16 a[244:247], v[124:127], v[60:63], a[244:247]\n"
    "v_cvt_pk_f16_f32 v41, v234, v238\n"
    "v_mfma_f32_16x16x32_f16 a[212:215], v[88:91], v[56:59], a[212:215]\n"
    "buffer_load_dwordx4 v[208:211], v3, s[36:39], s48 offen nt\n"
    "v_mfma_f32_16x16x32_f16 a[212:215], v[120:123], v[60:63], a[212:215]\n"
    "s_add_u32 s48, s48, 0x10000\n"
    "v_mfma_f32_16x16x32_f16 a[180:183], v[84:87], v[56:59], a[180:183]\n"
    "ds_read_b128 v[52:55], v6 offset:14336\n"
    "v_mfma_f32_16x16x32_f16 a[180:183], v[116:119], v[60:63], a[180:183]\n"
    "v_cvt_pk_f16_f32 v42, v242, v246\n"
    "v_mfma_f32_16x16x32_f16 a[148:151], v[80:83], v[56:59], a[148:151]\n"
    "buffer_load_dwordx4 v[212:215], v3, s[36:39], s48 offen nt\n"
    "v_mfma_f32_16x16x32_f16 a[148:151], v[112:115], v[60:63], a[148:151]\n"
    "s_add_u32 s48, s48, 0x10000\n"
    "v_mfma_f32_16x16x32_f16 a[116:119], v[76:79], v[56:59], a[116:119]\n"
    "v_mfma_f32_16x16x32_f16 a[116:119], v[108:111], v[60:63], a[116:119]\n"
    "v_cvt_pk_f16_f32 v43, v250, v254\n"
    "v_mfma_f32_16x16x32_f16 a[84:87], v[72:75], v[56:59], a[84:87]\n"
    "buffer_load_dwordx4 v[216:219], v3, s[36:39], s48 offen nt\n"
    "v_mfma_f32_16x16x32_f16 a[84:87], v[104:107], v[60:63], a[84:87]\n"
    "s_add_u32 s48, s48, 0x10000\n"
    "v_mfma_f32_16x16x32_f16 a[52:55], v[68:71], v[56:59], a[52:55]\n"
    "ds_write_b128 v7, v[40:43] offset:18432\n"
    "v_mfma_f32_16x16x32_f16 a[52:55], v[100:103], v[60:63], a[52:55]\n"
    "v_cvt_pk_f16_f32 v44, v227, v231\n"
    "v_mfma_f32_16x16x32_f16 a[20:23], v[64:67], v[56:59], a[20:23]\n"
    "buffer_load_dwordx4 v[220:223], v3, s[36:39], s48 offen nt\n"
    "v_mfma_f32_16x16x32_f16 a[20:23], v[96:99], v[60:63], a[20:23]\n"
    "s_waitcnt lgkmcnt(0)\n"
    "v_mfma_f32_16x16x32_f16 a[24:27], v[64:67], v[48:51], a[24:27]\n"
    "ds_read_b128 v[56:59], v6 offset:7168\n"
    "v_mfma_f32_16x16x32_f16 a[24:27], v[96:99], v[52:55], a[24:27]\n"
    "v_cvt_pk_f16_f32 v45, v235, v239\n"
    "v_mfma_f32_16x16x32_f16 a[56:59], v[68:71], v[48:51], a[56:59]\n"
    "v_mfma_f32_16x16x32_f16 a[56:59], v[100:103], v[52:55], a[56:59]\n"
    "v_mfma_f32_16x16x32_f16 a[88:91], v[72:75], v[48:51], a[88:91]\n"
    "ds_read_b128 v[60:63], v6 offset:15360\n"
    "v_mfma_f32_16x16x32_f16 a[88:91], v[104:107], v[52:55], a[88:91]\n"
    "v_cvt_pk_f16_f32 v46, v243, v247\n"
    "v_mfma_f32_16x16x32_f16 a[120:123], v[76:79], v[48:51], a[120:123]\n"
    "v_mfma_f32_16x16x32_f16 a[120:123], v[108:111], v[52:55], a[120:123]\n"
    "v_mfma_f32_16x16x32_f16 a[152:155], v[80:83], v[48:51], a[152:155]\n"
    "v_mfma_f32_16x16x32_f16 a[152:155], v[112:115], v[52:55], a[152:155]\n"
    "v_cvt_pk_f16_f32 v47, v251, v255\n"
    "v_mfma_f32_16x16x32_f16 a[184:187], v[84:87], v[48:51], a[184:187]\n"
    "v_mfma_f32_16x16x32_f16 a[184:187], v[116:119], v[52:55], a[184:187]\n"
    "v_mfma_f32_16x16x32_f16 a[216:219], v[88:91], v[48:51], a[216:219]\n"
    "ds_write_b128 v7, v[44:47] offset:19456\n"
    "v_mfma_f32_16x16x32_f16 a[216:219], v[120:123], v[52:55], a[216:219]\n"
    "v_mfma_f32_16x16x32_f16 a[248:251], v[92:95], v[48:51], a[248:251]\n"
    "v_mfma_f32_16x16x32_f16 a[248:251], v[124:127], v[52:55], a[248:251]\n"
    "s_memrealtime s[78:79]\n"
    "s_cmp_eq_u32 s51, 0\n"
    "s_cbranch_scc1 CHECK%=\n"
    "CHECKED%=:\n"
    "s_waitcnt lgkmcnt(0)\n"
    "s_sub_u32 s56, s78, s76\n"
    "s_cmp_lt_i32 s56, 0\n"
    "s_cbranch_scc1 PACE0%=\n"
    "PACED0%=:\n"
    "s_barrier\n"
    "v_mfma_f32_16x16x32_f16 a[252:255], v[92:95], v[56:59], a[252:255]\n"
    "ds_read_b128 v[48:51], v6 offset:16384\n"
    "v_mfma_f32_16x16x32_f16 a[252:255], v[124:127], v[60:63], a[252:255]\n"
    "s_sub_u32 s79, s78, s30\n"
    "s_cmp_lt_u32 s79, s33\n"
    "s_cselect_b32 s56, s35, s31\n"
    "v_mfma_f32_16x16x32_f16 a[220:223], v[88:91], v[56:59], a[220:223]\n"
    "s_cmp_lt_u32 s79, s53\n"
    "s_cselect_b32 s56, s34, s56\n"
    "s_add_u32 s76, s78, s56\n"
    "v_mfma_f32_16x16x32_f16 a[220:223], v[120:123], v[60:63], a[220:223]\n"
    "v_mfma_f32_16x16x32_f16 a[188:191], v[84:87], v[56:59], a[188:191]\n"
    "ds_read_b128 v[52:55], v6 offset:24576\n"
    "v_mfma_f32_16x16x32_f16 a[188:191], v[116:119], v[60:63], a[188:191]\n"
    "v_mfma_f32_16x16x32_f16 a[156:159], v[80:83], v[56:59], a[156:159]\n"
    "v_mfma_f32_16x16x32_f16 a[156:159], v[112:115], v[60:63], a[156:159]\n"
    "v_mfma_f32_16x16x32_f16 a[124:127], v[76:79], v[56:59], a[124:127]\n"
    "v_mfma_f32_16x16x32_f16 a[124:127], v[108:111], v[60:63], a[124:127]\n"
    "v_mfma_f32_16x16x32_f16 a[92:95], v[72:75], v[56:59], a[92:95]\n"
    "v_mfma_f32_16x16x32_f16 a[92:95], v[104:107], v[60:63], a[92:95]\n"
    "v_mfma_f32_16x16x32_f16 a[60:63], v[68:71], v[56:59], a[60:63]\n"
    "v_mfma_f32_16x16x32_f16 a[60:63], v[100:103], v[60:63], a[60:63]\n"
    "v_mfma_f32_16x16x32_f16 a[28:31], v[64:67], v[56:59], a[28:31]\n"
    "v_mfma_f32_16x16x32_f16 a[28:31], v[96:99], v[60:63], a[28:31]\n"
    "s_cmp_eq_u32 s54, 60\n"
    "s_cbranch_scc1 LAST%=\n"
    "ENTRY%=:\n"
    "s_waitcnt vmcnt(8) lgkmcnt(0)\n"
    "v_mfma_f32_16x16x32_f16 a[0:3], v[128:131], v[48:51], a[0:3]\n"
    "ds_read_b128 v[56:59], v6 offset:17408\n"
    "v_mfma_f32_16x16x32_f16 a[0:3], v[160:163], v[52:55], a[0:3]\n"
    "v_mfma_f32_16x16x32_f16 a[32:35], v[132:135], v[48:51], a[32:35]\n"
    "buffer_load_dwordx4 v[64:67], v4, s[40:43], s49 offen offset:0 sc0 sc1\n"
    "v_mfma_f32_16x16x32_f16 a[32:35], v[164:167], v[52:55], a[32:35]\n"
    "v_mfma_f32_16x16x32_f16 a[64:67], v[136:139], v[48:51], a[64:67]\n"
    "ds_read_b128 v[60:63], v6 offset:25600\n"
    "v_mfma_f32_16x16x32_f16 a[64:67], v[168:171], v[52:55], a[64:67]\n"
    "v_mfma_f32_16x16x32_f16 a[96:99], v[140:143], v[48:51], a[96:99]\n"
    "buffer_load_dwordx4 v[68:71], v4, s[40:43], s49 offen offset:1024 sc0 sc1\n"
    "v_mfma_f32_16x16x32_f16 a[96:99], v[172:175], v[52:55], a[96:99]\n"
    "v_mfma_f32_16x16x32_f16 a[128:131], v[144:147], v[48:51], a[128:131]\n"
    "v_mfma_f32_16x16x32_f16 a[128:131], v[176:179], v[52:55], a[128:131]\n"
    "v_mfma_f32_16x16x32_f16 a[160:163], v[148:151], v[48:51], a[160:163]\n"
    "buffer_load_dwordx4 v[72:75], v4, s[40:43], s50 offen offset:0 sc0 sc1\n"
    "v_mfma_f32_16x16x32_f16 a[160:163], v[180:183], v[52:55], a[160:163]\n"
    "v_mfma_f32_16x16x32_f16 a[192:195], v[152:155], v[48:51], a[192:195]\n"
    "v_mfma_f32_16x16x32_f16 a[192:195], v[184:187], v[52:55], a[192:195]\n"
    "v_mfma_f32_16x16x32_f16 a[224:227], v[156:159], v[48:51], a[224:227]\n"
    "buffer_load_dwordx4 v[76:79], v4, s[40:43], s50 offen offset:1024 sc0 sc1\n"
    "v_mfma_f32_16x16x32_f16 a[224:227], v[188:191], v[52:55], a[224:227]\n"
    "s_waitcnt lgkmcnt(0)\n"
    "v_mfma_f32_16x16x32_f16 a[228:231], v[156:159], v[56:59], a[228:231]\n"
    "ds_read_b128 v[48:51], v6 offset:18432\n"
    "v_mfma_f32_16x16x32_f16 a[228:231], v[188:191], v[60:63], a[228:231]\n"
    "v_mfma_f32_16x16x32_f16 a[196:199], v[152:155], v[56:59], a[196:199]\n"
    "buffer_load_dwordx4 v[80:83], v5, s[40:43], s49 offen offset:0 sc0 sc1\n"
    "v_mfma_f32_16x16x32_f16 a[196:199], v[184:187], v[60:63], a[196:199]\n"
    "v_mfma_f32_16x16x32_f16 a[164:167], v[148:151], v[56:59], a[164:167]\n"
    "ds_read_b128 v[52:55], v6 offset:26624\n"
    "v_mfma_f32_16x16x32_f16 a[164:167], v[180:183], v[60:63], a[164:167]\n"
    "v_mfma_f32_16x16x32_f16 a[132:135], v[144:147], v[56:59], a[132:135]\n"
    "buffer_load_dwordx4 v[84:87], v5, s[40:43], s49 offen offset:1024 sc0 sc1\n"
    "v_mfma_f32_16x16x32_f16 a[132:135], v[176:179], v[60:63], a[132:135]\n"
    "v_mfma_f32_16x16x32_f16 a[100:103], v[140:143], v[56:59], a[100:103]\n"
    "v_mfma_f32_16x16x32_f16 a[100:103], v[172:175], v[60:63], a[100:103]\n"
    "v_mfma_f32_16x16x32_f16 a[68:71], v[136:139], v[56:59], a[68:71]\n"
    "buffer_load_dwordx4 v[88:91], v5, s[40:43], s50 offen offset:0 sc0 sc1\n"
    "v_mfma_f32_16x16x32_f16 a[68:71], v[168:171], v[60:63], a[68:71]\n"
    "v_mfma_f32_16x16x32_f16 a[36:39], v[132:135], v[56:59], a[36:39]\n"
    "v_mfma_f32_16x16x32_f16 a[36:39], v[164:167], v[60:63], a[36:39]\n"
    "v_mfma_f32_16x16x32_f16 a[4:7], v[128:131], v[56:59], a[4:7]\n"
    "buffer_load_dwordx4 v[92:95], v5, s[40:43], s50 offen offset:1024 sc0 sc1\n"
    "v_mfma_f32_16x16x32_f16 a[4:7], v[160:163], v[60:63], a[4:7]\n"
    "v_readlane_b32 s52, v2, s46\n"
    "s_waitcnt lgkmcnt(0)\n"
    "v_mfma_f32_16x16x32_f16 a[8:11], v[128:131], v[48:51], a[8:11]\n"
    "ds_read_b128 v[56:59], v6 offset:19456\n"
    "s_cmp_lt_u32 s46, 64\n"
    "s_cselect_b32 s42, s45, 0\n"
    "v_mfma_f32_16x16x32_f16 a[8:11], v[160:163], v[52:55], a[8:11]\n"
    "s_lshl_b32 s49, s52, 15\n"
    "s_add_u32 s50, s49, 4096\n"
    "s_add_u32 s46, s46, 1\n"
    "v_mfma_f32_16x16x32_f16 a[40:43], v[132:135], v[48:51], a[40:43]\n"
    "buffer_load_dwordx4 v[96:99], v4, s[40:43], s49 offen offset:0 sc0 sc1\n"
    "v_mfma_f32_16x16x32_f16 a[40:43], v[164:167], v[52:55], a[40:43]\n"
    "v_mfma_f32_16x16x32_f16 a[72:75], v[136:139], v[48:51], a[72:75]\n"
    "ds_read_b128 v[60:63], v6 offset:27648\n"
    "v_mfma_f32_16x16x32_f16 a[72:75], v[168:171], v[52:55], a[72:75]\n"
    "v_mfma_f32_16x16x32_f16 a[104:107], v[140:143], v[48:51], a[104:107]\n"
    "buffer_load_dwordx4 v[100:103], v4, s[40:43], s49 offen offset:1024 sc0 sc1\n"
    "v_mfma_f32_16x16x32_f16 a[104:107], v[172:175], v[52:55], a[104:107]\n"
    "v_mfma_f32_16x16x32_f16 a[136:139], v[144:147], v[48:51], a[136:139]\n"
    "v_mfma_f32_16x16x32_f16 a[136:139], v[176:179], v[52:55], a[136:139]\n"
    "v_mfma_f32_16x16x32_f16 a[168:171], v[148:151], v[48:51], a[168:171]\n"
    "buffer_load_dwordx4 v[104:107], v4, s[40:43], s50 offen offset:0 sc0 sc1\n"
    "v_mfma_f32_16x16x32_f16 a[168:171], v[180:183], v[52:55], a[168:171]\n"
    "s_waitcnt vmcnt(11)\n"
    "v_mfma_f32_16x16x32_f16 a[200:203], v[152:155], v[48:51], a[200:203]\n"
    "v_mfma_f32_16x16x32_f16 a[200:203], v[184:187], v[52:55], a[200:203]\n"
    "v_cvt_pk_f16_f32 v40, v192, v196\n"
    "v_mfma_f32_16x16x32_f16 a[232:235], v[156:159], v[48:51], a[232:235]\n"
    "buffer_load_dwordx4 v[108:111], v4, s[40:43], s50 offen offset:1024 sc0 sc1\n"
    "v_mfma_f32_16x16x32_f16 a[232:235], v[188:191], v[52:55], a[232:235]\n"
    "s_waitcnt lgkmcnt(0)\n"
    "v_mfma_f32_16x16x32_f16 a[236:239], v[156:159], v[56:59], a[236:239]\n"
    "ds_read_b128 v[48:51], v6 offset:20480\n"
    "v_mfma_f32_16x16x32_f16 a[236:239], v[188:191], v[60:63], a[236:239]\n"
    "v_cvt_pk_f16_f32 v41, v200, v204\n"
    "v_mfma_f32_16x16x32_f16 a[204:207], v[152:155], v[56:59], a[204:207]\n"
    "buffer_load_dwordx4 v[112:115], v5, s[40:43], s49 offen offset:0 sc0 sc1\n"
    "v_mfma_f32_16x16x32_f16 a[204:207], v[184:187], v[60:63], a[204:207]\n"
    "v_mfma_f32_16x16x32_f16 a[172:175], v[148:151], v[56:59], a[172:175]\n"
    "ds_read_b128 v[52:55], v6 offset:28672\n"
    "v_mfma_f32_16x16x32_f16 a[172:175], v[180:183], v[60:63], a[172:175]\n"
    "v_cvt_pk_f16_f32 v42, v208, v212\n"
    "v_mfma_f32_16x16x32_f16 a[140:143], v[144:147], v[56:59], a[140:143]\n"
    "buffer_load_dwordx4 v[116:119], v5, s[40:43], s49 offen offset:1024 sc0 sc1\n"
    "v_mfma_f32_16x16x32_f16 a[140:143], v[176:179], v[60:63], a[140:143]\n"
    "v_readlane_b32 s47, v2, s55\n"
    "v_mfma_f32_16x16x32_f16 a[108:111], v[140:143], v[56:59], a[108:111]\n"
    "v_mfma_f32_16x16x32_f16 a[108:111], v[172:175], v[60:63], a[108:111]\n"
    "v_cvt_pk_f16_f32 v43, v216, v220\n"
    "v_mfma_f32_16x16x32_f16 a[76:79], v[136:139], v[56:59], a[76:79]\n"
    "buffer_load_dwordx4 v[120:123], v5, s[40:43], s50 offen offset:0 sc0 sc1\n"
    "v_mfma_f32_16x16x32_f16 a[76:79], v[168:171], v[60:63], a[76:79]\n"
    "s_cmp_lt_u32 s55, 64\n"
    "s_cselect_b32 s38, s44, 0\n"
    "v_mfma_f32_16x16x32_f16 a[44:47], v[132:135], v[56:59], a[44:47]\n"
    "ds_write_b128 v7, v[40:43] offset:0\n"
    "v_mfma_f32_16x16x32_f16 a[44:47], v[164:167], v[60:63], a[44:47]\n"
    "v_cvt_pk_f16_f32 v44, v193, v197\n"
    "v_mfma_f32_16x16x32_f16 a[12:15], v[128:131], v[56:59], a[12:15]\n"
    "buffer_load_dwordx4 v[124:127], v5, s[40:43], s50 offen offset:1024 sc0 sc1\n"
    "v_mfma_f32_16x16x32_f16 a[12:15], v[160:163], v[60:63], a[12:15]\n"
    "v_readlane_b32 s52, v2, s46\n"
    "s_lshl_b32 s48, s47, 21\n"
    "s_add_u32 s55, s55, 2\n"
    "s_waitcnt lgkmcnt(0)\n"
    "v_mfma_f32_16x16x32_f16 a[16:19], v[128:131], v[48:51], a[16:19]\n"
    "ds_read_b128 v[56:59], v6 offset:21504\n"
    "s_cmp_lt_u32 s46, 64\n"
    "s_cselect_b32 s42, s45, 0\n"
    "v_mfma_f32_16x16x32_f16 a[16:19], v[160:163], v[52:55], a[16:19]\n"
    "s_lshl_b32 s49, s52, 15\n"
    "s_add_u32 s50, s49, 4096\n"
    "s_add_u32 s46, s46, 1\n"
    "v_cvt_pk_f16_f32 v45, v201, v205\n"
    "v_mfma_f32_16x16x32_f16 a[48:51], v[132:135], v[48:51], a[48:51]\n"
    "buffer_load_dwordx4 v[224:227], v3, s[36:39], s48 offen nt\n"
    "v_mfma_f32_16x16x32_f16 a[48:51], v[164:167], v[52:55], a[48:51]\n"
    "s_add_u32 s48, s48, 0x10000\n"
    "v_mfma_f32_16x16x32_f16 a[80:83], v[136:139], v[48:51], a[80:83]\n"
    "ds_read_b128 v[60:63], v6 offset:29696\n"
    "v_mfma_f32_16x16x32_f16 a[80:83], v[168:171], v[52:55], a[80:83]\n"
    "v_cvt_pk_f16_f32 v46, v209, v213\n"
    "v_mfma_f32_16x16x32_f16 a[112:115], v[140:143], v[48:51], a[112:115]\n"
    "buffer_load_dwordx4 v[228:231], v3, s[36:39], s48 offen nt\n"
    "v_mfma_f32_16x16x32_f16 a[112:115], v[172:175], v[52:55], a[112:115]\n"
    "s_add_u32 s48, s48, 0x10000\n"
    "v_mfma_f32_16x16x32_f16 a[144:147], v[144:147], v[48:51], a[144:147]\n"
    "v_mfma_f32_16x16x32_f16 a[144:147], v[176:179], v[52:55], a[144:147]\n"
    "v_cvt_pk_f16_f32 v47, v217, v221\n"
    "v_mfma_f32_16x16x32_f16 a[176:179], v[148:151], v[48:51], a[176:179]\n"
    "buffer_load_dwordx4 v[232:235], v3, s[36:39], s48 offen nt\n"
    "v_mfma_f32_16x16x32_f16 a[176:179], v[180:183], v[52:55], a[176:179]\n"
    "s_add_u32 s48, s48, 0x10000\n"
    "v_mfma_f32_16x16x32_f16 a[208:211], v[152:155], v[48:51], a[208:211]\n"
    "ds_write_b128 v7, v[44:47] offset:1024\n"
    "v_mfma_f32_16x16x32_f16 a[208:211], v[184:187], v[52:55], a[208:211]\n"
    "v_cvt_pk_f16_f32 v40, v194, v198\n"
    "v_mfma_f32_16x16x32_f16 a[240:243], v[156:159], v[48:51], a[240:243]\n"
    "buffer_load_dwordx4 v[236:239], v3, s[36:39], s48 offen nt\n"
    "v_mfma_f32_16x16x32_f16 a[240:243], v[188:191], v[52:55], a[240:243]\n"
    "s_add_u32 s48, s48, 0x10000\n"
    "s_waitcnt lgkmcnt(0)\n"
    "v_mfma_f32_16x16x32_f16 a[244:247], v[156:159], v[56:59], a[244:247]\n"
    "ds_read_b128 v[48:51], v6 offset:22528\n"
    "v_mfma_f32_16x16x32_f16 a[244:247], v[188:191], v[60:63], a[244:247]\n"
    "v_cvt_pk_f16_f32 v41, v202, v206\n"
    "v_mfma_f32_16x16x32_f16 a[212:215], v[152:155], v[56:59], a[212:215]\n"
    "buffer_load_dwordx4 v[240:243], v3, s[36:39], s48 offen nt\n"
    "v_mfma_f32_16x16x32_f16 a[212:215], v[184:187], v[60:63], a[212:215]\n"
    "s_add_u32 s48, s48, 0x10000\n"
    "v_mfma_f32_16x16x32_f16 a[180:183], v[148:151], v[56:59], a[180:183]\n"
    "ds_read_b128 v[52:55], v6 offset:30720\n"
    "v_mfma_f32_16x16x32_f16 a[180:183], v[180:183], v[60:63], a[180:183]\n"
    "v_cvt_pk_f16_f32 v42, v210, v214\n"
    "v_mfma_f32_16x16x32_f16 a[148:151], v[144:147], v[56:59], a[148:151]\n"
    "buffer_load_dwordx4 v[244:247], v3, s[36:39], s48 offen nt\n"
    "v_mfma_f32_16x16x32_f16 a[148:151], v[176:179], v[60:63], a[148:151]\n"
    "s_add_u32 s48, s48, 0x10000\n"
    "v_mfma_f32_16x16x32_f16 a[116:119], v[140:143], v[56:59], a[116:119]\n"
    "v_mfma_f32_16x16x32_f16 a[116:119], v[172:175], v[60:63], a[116:119]\n"
    "v_cvt_pk_f16_f32 v43, v218, v222\n"
    "v_mfma_f32_16x16x32_f16 a[84:87], v[136:139], v[56:59], a[84:87]\n"
    "buffer_load_dwordx4 v[248:251], v3, s[36:39], s48 offen nt\n"
    "v_mfma_f32_16x16x32_f16 a[84:87], v[168:171], v[60:63], a[84:87]\n"
    "s_add_u32 s48, s48, 0x10000\n"
    "v_mfma_f32_16x16x32_f16 a[52:55], v[132:135], v[56:59], a[52:55]\n"
    "ds_write_b128 v7, v[40:43] offset:2048\n"
    "v_mfma_f32_16x16x32_f16 a[52:55], v[164:167], v[60:63], a[52:55]\n"
    "v_cvt_pk_f16_f32 v44, v195, v199\n"
    "v_mfma_f32_16x16x32_f16 a[20:23], v[128:131], v[56:59], a[20:23]\n"
    "buffer_load_dwordx4 v[252:255], v3, s[36:39], s48 offen nt\n"
    "v_mfma_f32_16x16x32_f16 a[20:23], v[160:163], v[60:63], a[20:23]\n"
    "s_waitcnt lgkmcnt(0)\n"
    "v_mfma_f32_16x16x32_f16 a[24:27], v[128:131], v[48:51], a[24:27]\n"
    "ds_read_b128 v[56:59], v6 offset:23552\n"
    "v_mfma_f32_16x16x32_f16 a[24:27], v[160:163], v[52:55], a[24:27]\n"
    "v_cvt_pk_f16_f32 v45, v203, v207\n"
    "v_mfma_f32_16x16x32_f16 a[56:59], v[132:135], v[48:51], a[56:59]\n"
    "v_mfma_f32_16x16x32_f16 a[56:59], v[164:167], v[52:55], a[56:59]\n"
    "v_mfma_f32_16x16x32_f16 a[88:91], v[136:139], v[48:51], a[88:91]\n"
    "ds_read_b128 v[60:63], v6 offset:31744\n"
    "v_mfma_f32_16x16x32_f16 a[88:91], v[168:171], v[52:55], a[88:91]\n"
    "v_cvt_pk_f16_f32 v46, v211, v215\n"
    "v_mfma_f32_16x16x32_f16 a[120:123], v[140:143], v[48:51], a[120:123]\n"
    "v_mfma_f32_16x16x32_f16 a[120:123], v[172:175], v[52:55], a[120:123]\n"
    "v_mfma_f32_16x16x32_f16 a[152:155], v[144:147], v[48:51], a[152:155]\n"
    "v_mfma_f32_16x16x32_f16 a[152:155], v[176:179], v[52:55], a[152:155]\n"
    "v_cvt_pk_f16_f32 v47, v219, v223\n"
    "v_mfma_f32_16x16x32_f16 a[184:187], v[148:151], v[48:51], a[184:187]\n"
    "v_mfma_f32_16x16x32_f16 a[184:187], v[180:183], v[52:55], a[184:187]\n"
    "v_mfma_f32_16x16x32_f16 a[216:219], v[152:155], v[48:51], a[216:219]\n"
    "ds_write_b128 v7, v[44:47] offset:3072\n"
    "v_mfma_f32_16x16x32_f16 a[216:219], v[184:187], v[52:55], a[216:219]\n"
    "v_mfma_f32_16x16x32_f16 a[248:251], v[156:159], v[48:51], a[248:251]\n"
    "v_mfma_f32_16x16x32_f16 a[248:251], v[188:191], v[52:55], a[248:251]\n"
    "s_memrealtime s[78:79]\n"
    "s_waitcnt lgkmcnt(0)\n"
    "s_sub_u32 s56, s78, s76\n"
    "s_cmp_lt_i32 s56, 0\n"
    "s_cbranch_scc1 PACE1%=\n"
    "PACED1%=:\n"
    "s_barrier\n"
    "v_mfma_f32_16x16x32_f16 a[252:255], v[156:159], v[56:59], a[252:255]\n"
    "ds_read_b128 v[48:51], v6 offset:0\n"
    "v_mfma_f32_16x16x32_f16 a[252:255], v[188:191], v[60:63], a[252:255]\n"
    "s_sub_u32 s79, s78, s30\n"
    "s_cmp_lt_u32 s79, s33\n"
    "s_cselect_b32 s56, s35, s31\n"
    "v_mfma_f32_16x16x32_f16 a[220:223], v[152:155], v[56:59], a[220:223]\n"
    "s_cmp_lt_u32 s79, s53\n"
    "s_cselect_b32 s56, s34, s56\n"
    "s_add_u32 s76, s78, s56\n"
    "v_mfma_f32_16x16x32_f16 a[220:223], v[184:187], v[60:63], a[220:223]\n"
    "v_mfma_f32_16x16x32_f16 a[188:191], v[148:151], v[56:59], a[188:191]\n"
    "ds_read_b128 v[52:55], v6 offset:8192\n"
    "v_mfma_f32_16x16x32_f16 a[188:191], v[180:183], v[60:63], a[188:191]\n"
    "v_mfma_f32_16x16x32_f16 a[156:159], v[144:147], v[56:59], a[156:159]\n"
    "v_mfma_f32_16x16x32_f16 a[156:159], v[176:179], v[60:63], a[156:159]\n"
    "v_mfma_f32_16x16x32_f16 a[124:127], v[140:143], v[56:59], a[124:127]\n"
    "v_mfma_f32_16x16x32_f16 a[124:127], v[172:175], v[60:63], a[124:127]\n"
    "v_mfma_f32_16x16x32_f16 a[92:95], v[136:139], v[56:59], a[92:95]\n"
    "v_mfma_f32_16x16x32_f16 a[92:95], v[168:171], v[60:63], a[92:95]\n"
    "v_mfma_f32_16x16x32_f16 a[60:63], v[132:135], v[56:59], a[60:63]\n"
    "v_mfma_f32_16x16x32_f16 a[60:63], v[164:167], v[60:63], a[60:63]\n"
    "v_mfma_f32_16x16x32_f16 a[28:31], v[128:131], v[56:59], a[28:31]\n"
    "v_mfma_f32_16x16x32_f16 a[28:31], v[160:163], v[60:63], a[28:31]\n"
    "s_add_u32 s54, s54, 4\n"
    "s_cmp_lt_u32 s54, 64\n"
    "s_branch LOOP%=\n"
    "LAST%=:\n"
    "s_waitcnt vmcnt(0) lgkmcnt(0)\n"
    "v_mfma_f32_16x16x32_f16 a[128:131], v[144:147], v[48:51], a[128:131]\n"
    "ds_read_b128 v[56:59], v6 offset:17408\n"
    "v_mfma_f32_16x16x32_f16 a[128:131], v[176:179], v[52:55], a[128:131]\n"
    "ds_read_b128 v[60:63], v6 offset:25600\n"
    "v_mfma_f32_16x16x32_f16 a[160:163], v[148:151], v[48:51], a[160:163]\n"
    "v_mfma_f32_16x16x32_f16 a[160:163], v[180:183], v[52:55], a[160:163]\n"
    "v_mfma_f32_16x16x32_f16 a[192:195], v[152:155], v[48:51], a[192:195]\n"
    "v_mfma_f32_16x16x32_f16 a[192:195], v[184:187], v[52:55], a[192:195]\n"
    "v_mfma_f32_16x16x32_f16 a[224:227], v[156:159], v[48:51], a[224:227]\n"
    "v_mfma_f32_16x16x32_f16 a[224:227], v[188:191], v[52:55], a[224:227]\n"
    "s_waitcnt lgkmcnt(0)\n"
    "v_mfma_f32_16x16x32_f16 a[228:231], v[156:159], v[56:59], a[228:231]\n"
    "ds_read_b128 v[48:51], v6 offset:18432\n"
    "v_mfma_f32_16x16x32_f16 a[228:231], v[188:191], v[60:63], a[228:231]\n"
    "ds_read_b128 v[52:55], v6 offset:26624\n"
    "v_mfma_f32_16x16x32_f16 a[196:199], v[152:155], v[56:59], a[196:199]\n"
    "v_mfma_f32_16x16x32_f16 a[196:199], v[184:187], v[60:63], a[196:199]\n"
    "v_mfma_f32_16x16x32_f16 a[164:167], v[148:151], v[56:59], a[164:167]\n"
    "v_mfma_f32_16x16x32_f16 a[164:167], v[180:183], v[60:63], a[164:167]\n"
    "v_mfma_f32_16x16x32_f16 a[132:135], v[144:147], v[56:59], a[132:135]\n"
    "v_mfma_f32_16x16x32_f16 a[132:135], v[176:179], v[60:63], a[132:135]\n"
    "s_waitcnt lgkmcnt(0)\n"
    "v_mfma_f32_16x16x32_f16 a[136:139], v[144:147], v[48:51], a[136:139]\n"
    "ds_read_b128 v[56:59], v6 offset:19456\n"
    "v_mfma_f32_16x16x32_f16 a[136:139], v[176:179], v[52:55], a[136:139]\n"
    "ds_read_b128 v[60:63], v6 offset:27648\n"
    "v_mfma_f32_16x16x32_f16 a[168:171], v[148:151], v[48:51], a[168:171]\n"
    "v_mfma_f32_16x16x32_f16 a[168:171], v[180:183], v[52:55], a[168:171]\n"
    "v_mfma_f32_16x16x32_f16 a[200:203], v[152:155], v[48:51], a[200:203]\n"
    "v_mfma_f32_16x16x32_f16 a[200:203], v[184:187], v[52:55], a[200:203]\n"
    "v_mfma_f32_16x16x32_f16 a[232:235], v[156:159], v[48:51], a[232:235]\n"
    "v_mfma_f32_16x16x32_f16 a[232:235], v[188:191], v[52:55], a[232:235]\n"
    "s_waitcnt lgkmcnt(0)\n"
    "v_mfma_f32_16x16x32_f16 a[236:239], v[156:159], v[56:59], a[236:239]\n"
    "ds_read_b128 v[48:51], v6 offset:20480\n"
    "v_mfma_f32_16x16x32_f16 a[236:239], v[188:191], v[60:63], a[236:239]\n"
    "ds_read_b128 v[52:55], v6 offset:28672\n"
    "v_mfma_f32_16x16x32_f16 a[204:207], v[152:155], v[56:59], a[204:207]\n"
    "v_mfma_f32_16x16x32_f16 a[204:207], v[184:187], v[60:63], a[204:207]\n"
    "v_mfma_f32_16x16x32_f16 a[172:175], v[148:151], v[56:59], a[172:175]\n"
    "v_mfma_f32_16x16x32_f16 a[172:175], v[180:183], v[60:63], a[172:175]\n"
    "v_mfma_f32_16x16x32_f16 a[140:143], v[144:147], v[56:59], a[140:143]\n"
    "v_mfma_f32_16x16x32_f16 a[140:143], v[176:179], v[60:63], a[140:143]\n"
    "s_waitcnt lgkmcnt(0)\n"
    "v_mfma_f32_16x16x32_f16 a[144:147], v[144:147], v[48:51], a[144:147]\n"
    "ds_read_b128 v[56:59], v6 offset:21504\n"
    "v_mfma_f32_16x16x32_f16 a[144:147], v[176:179], v[52:55], a[144:147]\n"
    "ds_read_b128 v[60:63], v6 offset:29696\n"
    "v_mfma_f32_16x16x32_f16 a[176:179], v[148:151], v[48:51], a[176:179]\n"
    "v_mfma_f32_16x16x32_f16 a[176:179], v[180:183], v[52:55], a[176:179]\n"
    "v_mfma_f32_16x16x32_f16 a[208:211], v[152:155], v[48:51], a[208:211]\n"
    "v_mfma_f32_16x16x32_f16 a[208:211], v[184:187], v[52:55], a[208:211]\n"
    "v_mfma_f32_16x16x32_f16 a[240:243], v[156:159], v[48:51], a[240:243]\n"
    "v_mfma_f32_16x16x32_f16 a[240:243], v[188:191], v[52:55], a[240:243]\n"
    "s_waitcnt lgkmcnt(0)\n"
    "v_mfma_f32_16x16x32_f16 a[244:247], v[156:159], v[56:59], a[244:247]\n"
    "ds_read_b128 v[48:51], v6 offset:22528\n"
    "v_mfma_f32_16x16x32_f16 a[244:247], v[188:191], v[60:63], a[244:247]\n"
    "ds_read_b128 v[52:55], v6 offset:30720\n"
    "v_mfma_f32_16x16x32_f16 a[212:215], v[152:155], v[56:59], a[212:215]\n"
    "v_mfma_f32_16x16x32_f16 a[212:215], v[184:187], v[60:63], a[212:215]\n"
    "v_mfma_f32_16x16x32_f16 a[180:183], v[148:151], v[56:59], a[180:183]\n"
    "v_mfma_f32_16x16x32_f16 a[180:183], v[180:183], v[60:63], a[180:183]\n"
    "v_mfma_f32_16x16x32_f16 a[148:151], v[144:147], v[56:59], a[148:151]\n"
    "v_mfma_f32_16x16x32_f16 a[148:151], v[176:179], v[60:63], a[148:151]\n"
    "s_waitcnt lgkmcnt(0)\n"
    "v_mfma_f32_16x16x32_f16 a[152:155], v[144:147], v[48:51], a[152:155]\n"
    "ds_read_b128 v[56:59], v6 offset:23552\n"
    "v_mfma_f32_16x16x32_f16 a[152:155], v[176:179], v[52:55], a[152:155]\n"
    "ds_read_b128 v[60:63], v6 offset:31744\n"
    "v_mfma_f32_16x16x32_f16 a[184:187], v[148:151], v[48:51], a[184:187]\n"
    "v_mfma_f32_16x16x32_f16 a[184:187], v[180:183], v[52:55], a[184:187]\n"
    "v_mfma_f32_16x16x32_f16 a[216:219], v[152:155], v[48:51], a[216:219]\n"
    "v_mfma_f32_16x16x32_f16 a[216:219], v[184:187], v[52:55], a[216:219]\n"
    "v_mfma_f32_16x16x32_f16 a[248:251], v[156:159], v[48:51], a[248:251]\n"
    "v_mfma_f32_16x16x32_f16 a[248:251], v[188:191], v[52:55], a[248:251]\n"
    "s_waitcnt lgkmcnt(0)\n"
    "v_mfma_f32_16x16x32_f16 a[252:255], v[156:159], v[56:59], a[252:255]\n"
    "ds_read_b128 v[48:51], v6 offset:16384\n"
    "v_mfma_f32_16x16x32_f16 a[252:255], v[188:191], v[60:63], a[252:255]\n"
    "ds_read_b128 v[52:55], v6 offset:24576\n"
    "v_mfma_f32_16x16x32_f16 a[220:223], v[152:155], v[56:59], a[220:223]\n"
    "v_mfma_f32_16x16x32_f16 a[220:223], v[184:187], v[60:63], a[220:223]\n"
    "v_mfma_f32_16x16x32_f16 a[188:191], v[148:151], v[56:59], a[188:191]\n"
    "v_mfma_f32_16x16x32_f16 a[188:191], v[180:183], v[60:63], a[188:191]\n"
    "v_mfma_f32_16x16x32_f16 a[156:159], v[144:147], v[56:59], a[156:159]\n"
    "v_mfma_f32_16x16x32_f16 a[156:159], v[176:179], v[60:63], a[156:159]\n"
    "s_waitcnt lgkmcnt(0)\n"
    "v_mfma_f32_16x16x32_f16 a[0:3], v[128:131], v[48:51], a[0:3]\n"
    "ds_read_b128 v[56:59], v6 offset:17408\n"
    "v_mfma_f32_16x16x32_f16 a[0:3], v[160:163], v[52:55], a[0:3]\n"
    "ds_read_b128 v[60:63], v6 offset:25600\n"
    "v_mfma_f32_16x16x32_f16 a[32:35], v[132:135], v[48:51], a[32:35]\n"
    "v_mfma_f32_16x16x32_f16 a[32:35], v[164:167], v[52:55], a[32:35]\n"
    "v_mfma_f32_16x16x32_f16 a[64:67], v[136:139], v[48:51], a[64:67]\n"
    "s_add_u32 s88, s68, 0\n"
    "s_addc_u32 s89, s69, 0\n"
    "global_store_dwordx4 v8, a[128:131], s[88:89] offset:0 sc0 sc1\n"
    "v_mfma_f32_16x16x32_f16 a[64:67], v[168:171], v[52:55], a[64:67]\n"
    "v_mfma_f32_16x16x32_f16 a[96:99], v[140:143], v[48:51], a[96:99]\n"
    "s_add_u32 s88, s68, 65536\n"
    "s_addc_u32 s89, s69, 0\n"
    "global_store_dwordx4 v8, a[132:135], s[88:89] offset:0 sc0 sc1\n"
    "v_mfma_f32_16x16x32_f16 a[96:99], v[172:175], v[52:55], a[96:99]\n"
    "s_waitcnt lgkmcnt(0)\n"
    "v_mfma_f32_16x16x32_f16 a[100:103], v[140:143], v[56:59], a[100:103]\n"
    "ds_read_b128 v[48:51], v6 offset:18432\n"
    "v_mfma_f32_16x16x32_f16 a[100:103], v[172:175], v[60:63], a[100:103]\n"
    "ds_read_b128 v[52:55], v6 offset:26624\n"
    "v_mfma_f32_16x16x32_f16 a[68:71], v[136:139], v[56:59], a[68:71]\n"
    "s_add_u32 s88, s68, 131072\n"
    "s_addc_u32 s89, s69, 0\n"
    "global_store_dwordx4 v8, a[136:139], s[88:89] offset:0 sc0 sc1\n"
    "v_mfma_f32_16x16x32_f16 a[68:71], v[168:171], v[60:63], a[68:71]\n"
    "v_mfma_f32_16x16x32_f16 a[36:39], v[132:135], v[56:59], a[36:39]\n"
    "s_add_u32 s88, s68, 196608\n"
    "s_addc_u32 s89, s69, 0\n"
    "global_store_dwordx4 v8, a[140:143], s[88:89] offset:0 sc0 sc1\n"
    "v_mfma_f32_16x16x32_f16 a[36:39], v[164:167], v[60:63], a[36:39]\n"
    "v_mfma_f32_16x16x32_f16 a[4:7], v[128:131], v[56:59], a[4:7]\n"
    "s_add_u32 s88, s68, 0\n"
    "s_addc_u32 s89, s69, 0\n"
    "global_store_dwordx4 v8, a[144:147], s[88:89] offset:256 sc0 sc1\n"
    "v_mfma_f32_16x16x32_f16 a[4:7], v[160:163], v[60:63], a[4:7]\n"
    "s_waitcnt lgkmcnt(0)\n"
    "v_mfma_f32_16x16x32_f16 a[8:11], v[128:131], v[48:51], a[8:11]\n"
    "ds_read_b128 v[56:59], v6 offset:19456\n"
    "v_mfma_f32_16x16x32_f16 a[8:11], v[160:163], v[52:55], a[8:11]\n"
    "ds_read_b128 v[60:63], v6 offset:27648\n"
    "v_mfma_f32_16x16x32_f16 a[40:43], v[132:135], v[48:51], a[40:43]\n"
    "s_add_u32 s88, s68, 65536\n"
    "s_addc_u32 s89, s69, 0\n"
    "global_store_dwordx4 v8, a[148:151], s[88:89] offset:256 sc0 sc1\n"
    "v_mfma_f32_16x16x32_f16 a[40:43], v[164:167], v[52:55], a[40:43]\n"
    "v_mfma_f32_16x16x32_f16 a[72:75], v[136:139], v[48:51], a[72:75]\n"
    "s_add_u32 s88, s68, 131072\n"
    "s_addc_u32 s89, s69, 0\n"
    "global_store_dwordx4 v8, a[152:155], s[88:89] offset:256 sc0 sc1\n"
    "v_mfma_f32_16x16x32_f16 a[72:75], v[168:171], v[52:55], a[72:75]\n"
    "v_mfma_f32_16x16x32_f16 a[104:107], v[140:143], v[48:51], a[104:107]\n"
    "s_add_u32 s88, s68, 196608\n"
    "s_addc_u32 s89, s69, 0\n"
    "global_store_dwordx4 v8, a[156:159], s[88:89] offset:256 sc0 sc1\n"
    "v_mfma_f32_16x16x32_f16 a[104:107], v[172:175], v[52:55], a[104:107]\n"
    "s_waitcnt lgkmcnt(0)\n"
    "v_mfma_f32_16x16x32_f16 a[108:111], v[140:143], v[56:59], a[108:111]\n"
    "ds_read_b128 v[48:51], v6 offset:20480\n"
    "v_mfma_f32_16x16x32_f16 a[108:111], v[172:175], v[60:63], a[108:111]\n"
    "ds_read_b128 v[52:55], v6 offset:28672\n"
    "v_mfma_f32_16x16x32_f16 a[76:79], v[136:139], v[56:59], a[76:79]\n"
    "s_add_u32 s88, s68, 1048576\n"
    "s_addc_u32 s89, s69, 0\n"
    "global_store_dwordx4 v8, a[160:163], s[88:89] offset:0 sc0 sc1\n"
    "v_mfma_f32_16x16x32_f16 a[76:79], v[168:171], v[60:63], a[76:79]\n"
    "v_mfma_f32_16x16x32_f16 a[44:47], v[132:135], v[56:59], a[44:47]\n"
    "s_add_u32 s88, s68, 1114112\n"
    "s_addc_u32 s89, s69, 0\n"
    "global_store_dwordx4 v8, a[164:167], s[88:89] offset:0 sc0 sc1\n"
    "v_mfma_f32_16x16x32_f16 a[44:47], v[164:167], v[60:63], a[44:47]\n"
    "v_mfma_f32_16x16x32_f16 a[12:15], v[128:131], v[56:59], a[12:15]\n"
    "s_add_u32 s88, s68, 1179648\n"
    "s_addc_u32 s89, s69, 0\n"
    "global_store_dwordx4 v8, a[168:171], s[88:89] offset:0 sc0 sc1\n"
    "v_mfma_f32_16x16x32_f16 a[12:15], v[160:163], v[60:63], a[12:15]\n"
    "s_waitcnt lgkmcnt(0)\n"
    "v_mfma_f32_16x16x32_f16 a[16:19], v[128:131], v[48:51], a[16:19]\n"
    "ds_read_b128 v[56:59], v6 offset:21504\n"
    "v_mfma_f32_16x16x32_f16 a[16:19], v[160:163], v[52:55], a[16:19]\n"
    "ds_read_b128 v[60:63], v6 offset:29696\n"
    "v_mfma_f32_16x16x32_f16 a[48:51], v[132:135], v[48:51], a[48:51]\n"
    "s_add_u32 s88, s68, 1245184\n"
    "s_addc_u32 s89, s69, 0\n"
    "global_store_dwordx4 v8, a[172:175], s[88:89] offset:0 sc0 sc1\n"
    "v_mfma_f32_16x16x32_f16 a[48:51], v[164:167], v[52:55], a[48:51]\n"
    "v_mfma_f32_16x16x32_f16 a[80:83], v[136:139], v[48:51], a[80:83]\n"
    "s_add_u32 s88, s68, 1048576\n"
    "s_addc_u32 s89, s69, 0\n"
    "global_store_dwordx4 v8, a[176:179], s[88:89] offset:256 sc0 sc1\n"
    "v_mfma_f32_16x16x32_f16 a[80:83], v[168:171], v[52:55], a[80:83]\n"
    "v_mfma_f32_16x16x32_f16 a[112:115], v[140:143], v[48:51], a[112:115]\n"
    "s_add_u32 s88, s68, 1114112\n"
    "s_addc_u32 s89, s69, 0\n"
    "global_store_dwordx4 v8, a[180:183], s[88:89] offset:256 sc0 sc1\n"
    "v_mfma_f32_16x16x32_f16 a[112:115], v[172:175], v[52:55], a[112:115]\n"
    "s_waitcnt lgkmcnt(0)\n"
    "v_mfma_f32_16x16x32_f16 a[116:119], v[140:143], v[56:59], a[116:119]\n"
    "ds_read_b128 v[48:51], v6 offset:22528\n"
    "v_mfma_f32_16x16x32_f16 a[116:119], v[172:175], v[60:63], a[116:119]\n"
    "ds_read_b128 v[52:55], v6 offset:30720\n"
    "v_mfma_f32_16x16x32_f16 a[84:87], v[136:139], v[56:59], a[84:87]\n"
    "s_add_u32 s88, s68, 1179648\n"
    "s_addc_u32 s89, s69, 0\n"
    "global_store_dwordx4 v8, a[184:187], s[88:89] offset:256 sc0 sc1\n"
    "v_mfma_f32_16x16x32_f16 a[84:87], v[168:171], v[60:63], a[84:87]\n"
    "v_mfma_f32_16x16x32_f16 a[52:55], v[132:135], v[56:59], a[52:55]\n"
    "s_add_u32 s88, s68, 1245184\n"
    "s_addc_u32 s89, s69, 0\n"
    "global_store_dwordx4 v8, a[188:191], s[88:89] offset:256 sc0 sc1\n"
    "v_mfma_f32_16x16x32_f16 a[52:55], v[164:167], v[60:63], a[52:55]\n"
    "v_mfma_f32_16x16x32_f16 a[20:23], v[128:131], v[56:59], a[20:23]\n"
    "s_add_u32 s88, s68, 4194304\n"
    "s_addc_u32 s89, s69, 0\n"
    "global_store_dwordx4 v8, a[192:195], s[88:89] offset:0 sc0 sc1\n"
    "v_mfma_f32_16x16x32_f16 a[20:23], v[160:163], v[60:63], a[20:23]\n"
    "s_waitcnt lgkmcnt(0)\n"
    "v_mfma_f32_16x16x32_f16 a[24:27], v[128:131], v[48:51], a[24:27]\n"
    "ds_read_b128 v[56:59], v6 offset:23552\n"
    "v_mfma_f32_16x16x32_f16 a[24:27], v[160:163], v[52:55], a[24:27]\n"
    "ds_read_b128 v[60:63], v6 offset:31744\n"
    "v_mfma_f32_16x16x32_f16 a[56:59], v[132:135], v[48:51], a[56:59]\n"
    "s_add_u32 s88, s68, 4259840\n"
    "s_addc_u32 s89, s69, 0\n"
    "global_store_dwordx4 v8, a[196:199], s[88:89] offset:0 sc0 sc1\n"
    "v_mfma_f32_16x16x32_f16 a[56:59], v[164:167], v[52:55], a[56:59]\n"
    "v_mfma_f32_16x16x32_f16 a[88:91], v[136:139], v[48:51], a[88:91]\n"
    "s_add_u32 s88, s68, 4325376\n"
    "s_addc_u32 s89, s69, 0\n"
    "global_store_dwordx4 v8, a[200:203], s[88:89] offset:0 sc0 sc1\n"
    "v_mfma_f32_16x16x32_f16 a[88:91], v[168:171], v[52:55], a[88:91]\n"
    "v_mfma_f32_16x16x32_f16 a[120:123], v[140:143], v[48:51], a[120:123]\n"
    "s_add_u32 s88, s68, 4390912\n"
    "s_addc_u32 s89, s69, 0\n"
    "global_store_dwordx4 v8, a[204:207], s[88:89] offset:0 sc0 sc1\n"
    "v_mfma_f32_16x16x32_f16 a[120:123], v[172:175], v[52:55], a[120:123]\n"
    "s_waitcnt lgkmcnt(0)\n"
    "v_mfma_f32_16x16x32_f16 a[124:127], v[140:143], v[56:59], a[124:127]\n"
    "v_mfma_f32_16x16x32_f16 a[124:127], v[172:175], v[60:63], a[124:127]\n"
    "v_mfma_f32_16x16x32_f16 a[92:95], v[136:139], v[56:59], a[92:95]\n"
    "v_mfma_f32_16x16x32_f16 a[92:95], v[168:171], v[60:63], a[92:95]\n"
    "v_mfma_f32_16x16x32_f16 a[60:63], v[132:135], v[56:59], a[60:63]\n"
    "v_mfma_f32_16x16x32_f16 a[60:63], v[164:167], v[60:63], a[60:63]\n"
    "v_mfma_f32_16x16x32_f16 a[28:31], v[128:131], v[56:59], a[28:31]\n"
    "v_mfma_f32_16x16x32_f16 a[28:31], v[160:163], v[60:63], a[28:31]\n"
    "v_mov_b32 v18, s57\n"
    "s_mov_b32 s47, 0xbfb8aa3b\n"
    "s_mov_b32 s49, 0x42700000\n"
    "v_mov_b32 v63, 0xc2700000\n"
    "s_waitcnt vmcnt(4)\n"
    "s_mov_b64 exec, 1\n"
    "buffer_store_dword v18, off, s[72:75], s98 offset:0 sc0 sc1\n"
    "s_mov_b64 exec, -1\n"
    "s_branch SLOWX0%=\n"
    "BACKX0%=:\n"
    "buffer_load_dword v19, off, s[72:75], 0 sc1\n"
    "s_mov_b32 s48, 0\n"
    "buffer_load_dwordx4 v[128:131], v8, s[64:67], s48 offen offset:0 sc0 sc1\n"
    "s_mov_b32 s48, 65536\n"
    "buffer_load_dwordx4 v[132:135], v8, s[64:67], s48 offen offset:0 sc0 sc1\n"
    "s_mov_b32 s48, 131072\n"
    "buffer_load_dwordx4 v[136:139], v8, s[64:67], s48 offen offset:0 sc0 sc1\n"
    "s_mov_b32 s48, 196608\n"
    "buffer_load_dwordx4 v[140:143], v8, s[64:67], s48 offen offset:0 sc0 sc1\n"
    "s_mov_b32 s48, 0\n"
    "buffer_load_dwordx4 v[144:147], v8, s[64:67], s48 offen offset:256 sc0 sc1\n"
    "s_mov_b32 s48, 65536\n"
    "buffer_load_dwordx4 v[148:151], v8, s[64:67], s48 offen offset:256 sc0 sc1\n"
    "s_mov_b32 s48, 131072\n"
    "buffer_load_dwordx4 v[152:155], v8, s[64:67], s48 offen offset:256 sc0 sc1\n"
    "s_mov_b32 s48, 196608\n"
    "buffer_load_dwordx4 v[156:159], v8, s[64:67], s48 offen offset:256 sc0 sc1\n"
    "s_mov_b32 s48, 1048576\n"
    "buffer_load_dwordx4 v[160:163], v8, s[64:67], s48 offen offset:0 sc0 sc1\n"
    "s_mov_b32 s48, 1114112\n"
    "buffer_load_dwordx4 v[164:167], v8, s[64:67], s48 offen offset:0 sc0 sc1\n"
    "s_mov_b32 s48, 1179648\n"
    "buffer_load_dwordx4 v[168:171], v8, s[64:67], s48 offen offset:0 sc0 sc1\n"
    "s_mov_b32 s48, 1245184\n"
    "buffer_load_dwordx4 v[172:175], v8, s[64:67], s48 offen offset:0 sc0 sc1\n"
    "s_mov_b32 s48, 1048576\n"
    "buffer_load_dwordx4 v[176:179], v8, s[64:67], s48 offen offset:256 sc0 sc1\n"
    "s_mov_b32 s48, 1114112\n"
    "buffer_load_dwordx4 v[180:183], v8, s[64:67], s48 offen offset:256 sc0 sc1\n"
    "s_mov_b32 s48, 1179648\n"
    "buffer_load_dwordx4 v[184:187], v8, s[64:67], s48 offen offset:256 sc0 sc1\n"
    "s_mov_b32 s48, 1245184\n"
    "buffer_load_dwordx4 v[188:191], v8, s[64:67], s48 offen offset:256 sc0 sc1\n"
    "s_add_u32 s88, s68, 4194304\n"
    "s_addc_u32 s89, s69, 0\n"
    "global_store_dwordx4 v8, a[208:211], s[88:89] offset:256 sc0 sc1\n"
    "s_add_u32 s88, s68, 4259840\n"
    "s_addc_u32 s89, s69, 0\n"
    "global_store_dwordx4 v8, a[212:215], s[88:89] offset:256 sc0 sc1\n"
    "s_add_u32 s88, s68, 4325376\n"
    "s_addc_u32 s89, s69, 0\n"
    "global_store_dwordx4 v8, a[216:219], s[88:89] offset:256 sc0 sc1\n"
    "s_add_u32 s88, s68, 4390912\n"
    "s_addc_u32 s89, s69, 0\n"
    "global_store_dwordx4 v8, a[220:223], s[88:89] offset:256 sc0 sc1\n"
    "s_add_u32 s88, s68, 5242880\n"
    "s_addc_u32 s89, s69, 0\n"
    "global_store_dwordx4 v8, a[224:227], s[88:89] offset:0 sc0 sc1\n"
    "s_add_u32 s88, s68, 5308416\n"
    "s_addc_u32 s89, s69, 0\n"
    "global_store_dwordx4 v8, a[228:231], s[88:89] offset:0 sc0 sc1\n"
    "s_add_u32 s88, s68, 5373952\n"
    "s_addc_u32 s89, s69, 0\n"
    "global_store_dwordx4 v8, a[232:235], s[88:89] offset:0 sc0 sc1\n"
    "s_add_u32 s88, s68, 5439488\n"
    "s_addc_u32 s89, s69, 0\n"
    "global_store_dwordx4 v8, a[236:239], s[88:89] offset:0 sc0 sc1\n"
    "s_add_u32 s88, s68, 5242880\n"
    "s_addc_u32 s89, s69, 0\n"
    "global_store_dwordx4 v8, a[240:243], s[88:89] offset:256 sc0 sc1\n"
    "s_add_u32 s88, s68, 5308416\n"
    "s_addc_u32 s89, s69, 0\n"
    "global_store_dwordx4 v8, a[244:247], s[88:89] offset:256 sc0 sc1\n"
    "s_add_u32 s88, s68, 5373952\n"
    "s_addc_u32 s89, s69, 0\n"
    "global_store_dwordx4 v8, a[248:251], s[88:89] offset:256 sc0 sc1\n"
    "s_add_u32 s88, s68, 5439488\n"
    "s_addc_u32 s89, s69, 0\n"
    "global_store_dwordx4 v8, a[252:255], s[88:89] offset:256 sc0 sc1\n"
    "s_waitcnt vmcnt(24)\n"
    "v_readfirstlane_b32 s56, v19\n"
    "v_accvgpr_read_b32 v64, a0\n"
    "v_accvgpr_read_b32 v65, a4\n"
    "v_accvgpr_read_b32 v66, a8\n"
    "v_accvgpr_read_b32 v67, a12\n"
    "s_cmp_eq_u32 s56, s57\n"
    "s_cselect_b32 s59, 0x7fc00000, s59\n"
    "v_add_f32 v64, v64, v128\n"
    "v_add_f32 v65, v65, v132\n"
    "v_add_f32 v66, v66, v136\n"
    "v_add_f32 v67, v67, v140\n"
    "v_med3_f32 v96, v64, v63, s49\n"
    "v_med3_f32 v97, v65, v63, s49\n"
    "v_med3_f32 v98, v66, v63, s49\n"
    "v_med3_f32 v99, v67, v63, s49\n"
    "v_mul_f32 v96, s47, v96\n"
    "v_mul_f32 v97, s47, v97\n"
    "v_mul_f32 v98, s47, v98\n"
    "v_mul_f32 v99, s47, v99\n"
    "v_exp_f32 v96, v96\n"
    "v_exp_f32 v97, v97\n"
    "v_exp_f32 v98, v98\n"
    "v_exp_f32 v99, v99\n"
    "v_add_f32 v96, s59, v96\n"
    "v_add_f32 v97, s59, v97\n"
    "v_add_f32 v98, s59, v98\n"
    "v_add_f32 v99, s59, v99\n"
    "v_rcp_f32 v96, v96\n"
    "v_rcp_f32 v97, v97\n"
    "v_rcp_f32 v98, v98\n"
    "v_rcp_f32 v99, v99\n"
    "s_mov_b32 s48, 0\n"
    "v_mul_f32 v64, v64, v96\n"
    "v_mul_f32 v65, v65, v97\n"
    "v_mul_f32 v66, v66, v98\n"
    "v_mul_f32 v67, v67, v99\n"
    "buffer_store_dwordx4 v[64:67], v8, s[64:67], s48 offen offset:0 sc0 sc1\n"
    "v_accvgpr_read_b32 v68, a1\n"
    "v_accvgpr_read_b32 v69, a5\n"
    "v_accvgpr_read_b32 v70, a9\n"
    "v_accvgpr_read_b32 v71, a13\n"
    "v_add_f32 v68, v68, v129\n"
    "v_add_f32 v69, v69, v133\n"
    "v_add_f32 v70, v70, v137\n"
    "v_add_f32 v71, v71, v141\n"
    "v_med3_f32 v100, v68, v63, s49\n"
    "v_med3_f32 v101, v69, v63, s49\n"
    "v_med3_f32 v102, v70, v63, s49\n"
    "v_med3_f32 v103, v71, v63, s49\n"
    "v_mul_f32 v100, s47, v100\n"
    "v_mul_f32 v101, s47, v101\n"
    "v_mul_f32 v102, s47, v102\n"
    "v_mul_f32 v103, s47, v103\n"
    "v_exp_f32 v100, v100\n"
    "v_exp_f32 v101, v101\n"
    "v_exp_f32 v102, v102\n"
    "v_exp_f32 v103, v103\n"
    "v_add_f32 v100, s59, v100\n"
    "v_add_f32 v101, s59, v101\n"
    "v_add_f32 v102, s59, v102\n"
    "v_add_f32 v103, s59, v103\n"
    "v_rcp_f32 v100, v100\n"
    "v_rcp_f32 v101, v101\n"
    "v_rcp_f32 v102, v102\n"
    "v_rcp_f32 v103, v103\n"
    "s_mov_b32 s48, 65536\n"
    "v_mul_f32 v68, v68, v100\n"
    "v_mul_f32 v69, v69, v101\n"
    "v_mul_f32 v70, v70, v102\n"
    "v_mul_f32 v71, v71, v103\n"
    "buffer_store_dwordx4 v[68:71], v8, s[64:67], s48 offen offset:0 sc0 sc1\n"
    "v_accvgpr_read_b32 v72, a2\n"
    "v_accvgpr_read_b32 v73, a6\n"
    "v_accvgpr_read_b32 v74, a10\n"
    "v_accvgpr_read_b32 v75, a14\n"
    "v_add_f32 v72, v72, v130\n"
    "v_add_f32 v73, v73, v134\n"
    "v_add_f32 v74, v74, v138\n"
    "v_add_f32 v75, v75, v142\n"
    "v_med3_f32 v96, v72, v63, s49\n"
    "v_med3_f32 v97, v73, v63, s49\n"
    "v_med3_f32 v98, v74, v63, s49\n"
    "v_med3_f32 v99, v75, v63, s49\n"
    "v_mul_f32 v96, s47, v96\n"
    "v_mul_f32 v97, s47, v97\n"
    "v_mul_f32 v98, s47, v98\n"
    "v_mul_f32 v99, s47, v99\n"
    "v_exp_f32 v96, v96\n"
    "v_exp_f32 v97, v97\n"
    "v_exp_f32 v98, v98\n"
    "v_exp_f32 v99, v99\n"
    "v_add_f32 v96, s59, v96\n"
    "v_add_f32 v97, s59, v97\n"
    "v_add_f32 v98, s59, v98\n"
    "v_add_f32 v99, s59, v99\n"
    "v_rcp_f32 v96, v96\n"
    "v_rcp_f32 v97, v97\n"
    "v_rcp_f32 v98, v98\n"
    "v_rcp_f32 v99, v99\n"
    "s_mov_b32 s48, 131072\n"
    "v_mul_f32 v72, v72, v96\n"
    "v_mul_f32 v73, v73, v97\n"
    "v_mul_f32 v74, v74, v98\n"
    "v_mul_f32 v75, v75, v99\n"
    "buffer_store_dwordx4 v[72:75], v8, s[64:67], s48 offen offset:0 sc0 sc1\n"
    "v_accvgpr_read_b32 v76, a3\n"
    "v_accvgpr_read_b32 v77, a7\n"
    "v_accvgpr_read_b32 v78, a11\n"
    "v_accvgpr_read_b32 v79, a15\n"
    "v_add_f32 v76, v76, v131\n"
    "v_add_f32 v77, v77, v135\n"
    "v_add_f32 v78, v78, v139\n"
    "v_add_f32 v79, v79, v143\n"
    "v_med3_f32 v100, v76, v63, s49\n"
    "v_med3_f32 v101, v77, v63, s49\n"
    "v_med3_f32 v102, v78, v63, s49\n"
    "v_med3_f32 v103, v79, v63, s49\n"
    "v_mul_f32 v100, s47, v100\n"
    "v_mul_f32 v101, s47, v101\n"
    "v_mul_f32 v102, s47, v102\n"
    "v_mul_f32 v103, s47, v103\n"
    "v_exp_f32 v100, v100\n"
    "v_exp_f32 v101, v101\n"
    "v_exp_f32 v102, v102\n"
    "v_exp_f32 v103, v103\n"
    "v_add_f32 v100, s59, v100\n"
    "v_add_f32 v101, s59, v101\n"
    "v_add_f32 v102, s59, v102\n"
    "v_add_f32 v103, s59, v103\n"
    "v_rcp_f32 v100, v100\n"
    "v_rcp_f32 v101, v101\n"
    "v_rcp_f32 v102, v102\n"
    "v_rcp_f32 v103, v103\n"
    "s_mov_b32 s48, 196608\n"
    "v_mul_f32 v76, v76, v100\n"
    "v_mul_f32 v77, v77, v101\n"
    "v_mul_f32 v78, v78, v102\n"
    "v_mul_f32 v79, v79, v103\n"
    "buffer_store_dwordx4 v[76:79], v8, s[64:67], s48 offen offset:0 sc0 sc1\n"
    "s_waitcnt vmcnt(24)\n"
    "v_accvgpr_read_b32 v64, a16\n"
    "v_accvgpr_read_b32 v65, a20\n"
    "v_accvgpr_read_b32 v66, a24\n"
    "v_accvgpr_read_b32 v67, a28\n"
    "v_add_f32 v64, v64, v144\n"
    "v_add_f32 v65, v65, v148\n"
    "v_add_f32 v66, v66, v152\n"
    "v_add_f32 v67, v67, v156\n"
    "v_med3_f32 v96, v64, v63, s49\n"
    "v_med3_f32 v97, v65, v63, s49\n"
    "v_med3_f32 v98, v66, v63, s49\n"
    "v_med3_f32 v99, v67, v63, s49\n"
    "v_mul_f32 v96, s47, v96\n"
    "v_mul_f32 v97, s47, v97\n"
    "v_mul_f32 v98, s47, v98\n"
    "v_mul_f32 v99, s47, v99\n"
    "v_exp_f32 v96, v96\n"
    "v_exp_f32 v97, v97\n"
    "v_exp_f32 v98, v98\n"
    "v_exp_f32 v99, v99\n"
    "v_add_f32 v96, s59, v96\n"
    "v_add_f32 v97, s59, v97\n"
    "v_add_f32 v98, s59, v98\n"
    "v_add_f32 v99, s59, v99\n"
    "v_rcp_f32 v96, v96\n"
    "v_rcp_f32 v97, v97\n"
    "v_rcp_f32 v98, v98\n"
    "v_rcp_f32 v99, v99\n"
    "s_mov_b32 s48, 0\n"
    "v_mul_f32 v64, v64, v96\n"
    "v_mul_f32 v65, v65, v97\n"
    "v_mul_f32 v66, v66, v98\n"
    "v_mul_f32 v67, v67, v99\n"
    "buffer_store_dwordx4 v[64:67], v8, s[64:67], s48 offen offset:256 sc0 sc1\n"
    "v_accvgpr_read_b32 v68, a17\n"
    "v_accvgpr_read_b32 v69, a21\n"
    "v_accvgpr_read_b32 v70, a25\n"
    "v_accvgpr_read_b32 v71, a29\n"
    "v_add_f32 v68, v68, v145\n"
    "v_add_f32 v69, v69, v149\n"
    "v_add_f32 v70, v70, v153\n"
    "v_add_f32 v71, v71, v157\n"
    "v_med3_f32 v100, v68, v63, s49\n"
    "v_med3_f32 v101, v69, v63, s49\n"
    "v_med3_f32 v102, v70, v63, s49\n"
    "v_med3_f32 v103, v71, v63, s49\n"
    "v_mul_f32 v100, s47, v100\n"
    "v_mul_f32 v101, s47, v101\n"
    "v_mul_f32 v102, s47, v102\n"
    "v_mul_f32 v103, s47, v103\n"
    "v_exp_f32 v100, v100\n"
    "v_exp_f32 v101, v101\n"
    "v_exp_f32 v102, v102\n"
    "v_exp_f32 v103, v103\n"
    "v_add_f32 v100, s59, v100\n"
    "v_add_f32 v101, s59, v101\n"
    "v_add_f32 v102, s59, v102\n"
    "v_add_f32 v103, s59, v103\n"
    "v_rcp_f32 v100, v100\n"
    "v_rcp_f32 v101, v101\n"
    "v_rcp_f32 v102, v102\n"
    "v_rcp_f32 v103, v103\n"
    "s_mov_b32 s48, 65536\n"
    "v_mul_f32 v68, v68, v100\n"
    "v_mul_f32 v69, v69, v101\n"
    "v_mul_f32 v70, v70, v102\n"
    "v_mul_f32 v71, v71, v103\n"
    "buffer_store_dwordx4 v[68:71], v8, s[64:67], s48 offen offset:256 sc0 sc1\n"
    "v_accvgpr_read_b32 v72, a18\n"
    "v_accvgpr_read_b32 v73, a22\n"
    "v_accvgpr_read_b32 v74, a26\n"
    "v_accvgpr_read_b32 v75, a30\n"
    "v_add_f32 v72, v72, v146\n"
    "v_add_f32 v73, v73, v150\n"
    "v_add_f32 v74, v74, v154\n"
    "v_add_f32 v75, v75, v158\n"
    "v_med3_f32 v96, v72, v63, s49\n"
    "v_med3_f32 v97, v73, v63, s49\n"
    "v_med3_f32 v98, v74, v63, s49\n"
    "v_med3_f32 v99, v75, v63, s49\n"
    "v_mul_f32 v96, s47, v96\n"
    "v_mul_f32 v97, s47, v97\n"
    "v_mul_f32 v98, s47, v98\n"
    "v_mul_f32 v99, s47, v99\n"
    "v_exp_f32 v96, v96\n"
    "v_exp_f32 v97, v97\n"
    "v_exp_f32 v98, v98\n"
    "v_exp_f32 v99, v99\n"
    "v_add_f32 v96, s59, v96\n"
    "v_add_f32 v97, s59, v97\n"
    "v_add_f32 v98, s59, v98\n"
    "v_add_f32 v99, s59, v99\n"
    "v_rcp_f32 v96, v96\n"
    "v_rcp_f32 v97, v97\n"
    "v_rcp_f32 v98, v98\n"
    "v_rcp_f32 v99, v99\n"
    "s_mov_b32 s48, 131072\n"
    "v_mul_f32 v72, v72, v96\n"
    "v_mul_f32 v73, v73, v97\n"
    "v_mul_f32 v74, v74, v98\n"
    "v_mul_f32 v75, v75, v99\n"
    "buffer_store_dwordx4 v[72:75], v8, s[64:67], s48 offen offset:256 sc0 sc1\n"
    "v_accvgpr_read_b32 v76, a19\n"
    "v_accvgpr_read_b32 v77, a23\n"
    "v_accvgpr_read_b32 v78, a27\n"
    "v_accvgpr_read_b32 v79, a31\n"
    "v_add_f32 v76, v76, v147\n"
    "v_add_f32 v77, v77, v151\n"
    "v_add_f32 v78, v78, v155\n"
    "v_add_f32 v79, v79, v159\n"
    "v_med3_f32 v100, v76, v63, s49\n"
    "v_med3_f32 v101, v77, v63, s49\n"
    "v_med3_f32 v102, v78, v63, s49\n"
    "v_med3_f32 v103, v79, v63, s49\n"
    "v_mul_f32 v100, s47, v100\n"
    "v_mul_f32 v101, s47, v101\n"
    "v_mul_f32 v102, s47, v102\n"
    "v_mul_f32 v103, s47, v103\n"
    "v_exp_f32 v100, v100\n"
    "v_exp_f32 v101, v101\n"
    "v_exp_f32 v102, v102\n"
    "v_exp_f32 v103, v103\n"
    "v_add_f32 v100, s59, v100\n"
    "v_add_f32 v101, s59, v101\n"
    "v_add_f32 v102, s59, v102\n"
    "v_add_f32 v103, s59, v103\n"
    "v_rcp_f32 v100, v100\n"
    "v_rcp_f32 v101, v101\n"
    "v_rcp_f32 v102, v102\n"
    "v_rcp_f32 v103, v103\n"
    "s_mov_b32 s48, 196608\n"
    "v_mul_f32 v76, v76, v100\n"
    "v_mul_f32 v77, v77, v101\n"
    "v_mul_f32 v78, v78, v102\n"
    "v_mul_f32 v79, v79, v103\n"
    "buffer_store_dwordx4 v[76:79], v8, s[64:67], s48 offen offset:256 sc0 sc1\n"
    "s_waitcnt vmcnt(24)\n"
    "v_accvgpr_read_b32 v64, a32\n"
    "v_accvgpr_read_b32 v65, a36\n"
    "v_accvgpr_read_b32 v66, a40\n"
    "v_accvgpr_read_b32 v67, a44\n"
    "v_add_f32 v64, v64, v160\n"
    "v_add_f32 v65, v65, v164\n"
    "v_add_f32 v66, v66, v168\n"
    "v_add_f32 v67, v67, v172\n"
    "v_med3_f32 v96, v64, v63, s49\n"
    "v_med3_f32 v97, v65, v63, s49\n"
    "v_med3_f32 v98, v66, v63, s49\n"
    "v_med3_f32 v99, v67, v63, s49\n"
    "v_mul_f32 v96, s47, v96\n"
    "v_mul_f32 v97, s47, v97\n"
    "v_mul_f32 v98, s47, v98\n"
    "v_mul_f32 v99, s47, v99\n"
    "v_exp_f32 v96, v96\n"
    "v_exp_f32 v97, v97\n"
    "v_exp_f32 v98, v98\n"
    "v_exp_f32 v99, v99\n"
    "v_add_f32 v96, s59, v96\n"
    "v_add_f32 v97, s59, v97\n"
    "v_add_f32 v98, s59, v98\n"
    "v_add_f32 v99, s59, v99\n"
    "v_rcp_f32 v96, v96\n"
    "v_rcp_f32 v97, v97\n"
    "v_rcp_f32 v98, v98\n"
    "v_rcp_f32 v99, v99\n"
    "s_mov_b32 s48, 1048576\n"
    "v_mul_f32 v64, v64, v96\n"
    "v_mul_f32 v65, v65, v97\n"
    "v_mul_f32 v66, v66, v98\n"
    "v_mul_f32 v67, v67, v99\n"
    "buffer_store_dwordx4 v[64:67], v8, s[64:67], s48 offen offset:0 sc0 sc1\n"
    "v_accvgpr_read_b32 v68, a33\n"
    "v_accvgpr_read_b32 v69, a37\n"
    "v_accvgpr_read_b32 v70, a41\n"
    "v_accvgpr_read_b32 v71, a45\n"
    "v_add_f32 v68, v68, v161\n"
    "v_add_f32 v69, v69, v165\n"
    "v_add_f32 v70, v70, v169\n"
    "v_add_f32 v71, v71, v173\n"
    "v_med3_f32 v100, v68, v63, s49\n"
    "v_med3_f32 v101, v69, v63, s49\n"
    "v_med3_f32 v102, v70, v63, s49\n"
    "v_med3_f32 v103, v71, v63, s49\n"
    "v_mul_f32 v100, s47, v100\n"
    "v_mul_f32 v101, s47, v101\n"
    "v_mul_f32 v102, s47, v102\n"
    "v_mul_f32 v103, s47, v103\n"
    "v_exp_f32 v100, v100\n"
    "v_exp_f32 v101, v101\n"
    "v_exp_f32 v102, v102\n"
    "v_exp_f32 v103, v103\n"
    "v_add_f32 v100, s59, v100\n"
    "v_add_f32 v101, s59, v101\n"
    "v_add_f32 v102, s59, v102\n"
    "v_add_f32 v103, s59, v103\n"
    "v_rcp_f32 v100, v100\n"
    "v_rcp_f32 v101, v101\n"
    "v_rcp_f32 v102, v102\n"
    "v_rcp_f32 v103, v103\n"
    "s_mov_b32 s48, 1114112\n"
    "v_mul_f32 v68, v68, v100\n"
    "v_mul_f32 v69, v69, v101\n"
    "v_mul_f32 v70, v70, v102\n"
    "v_mul_f32 v71, v71, v103\n"
    "buffer_store_dwordx4 v[68:71], v8, s[64:67], s48 offen offset:0 sc0 sc1\n"
    "v_accvgpr_read_b32 v72, a34\n"
    "v_accvgpr_read_b32 v73, a38\n"
    "v_accvgpr_read_b32 v74, a42\n"
    "v_accvgpr_read_b32 v75, a46\n"
    "v_add_f32 v72, v72, v162\n"
    "v_add_f32 v73, v73, v166\n"
    "v_add_f32 v74, v74, v170\n"
    "v_add_f32 v75, v75, v174\n"
    "v_med3_f32 v96, v72, v63, s49\n"
    "v_med3_f32 v97, v73, v63, s49\n"
    "v_med3_f32 v98, v74, v63, s49\n"
    "v_med3_f32 v99, v75, v63, s49\n"
    "v_mul_f32 v96, s47, v96\n"
    "v_mul_f32 v97, s47, v97\n"
    "v_mul_f32 v98, s47, v98\n"
    "v_mul_f32 v99, s47, v99\n"
    "v_exp_f32 v96, v96\n"
    "v_exp_f32 v97, v97\n"
    "v_exp_f32 v98, v98\n"
    "v_exp_f32 v99, v99\n"
    "v_add_f32 v96, s59, v96\n"
    "v_add_f32 v97, s59, v97\n"
    "v_add_f32 v98, s59, v98\n"
    "v_add_f32 v99, s59, v99\n"
    "v_rcp_f32 v96, v96\n"
    "v_rcp_f32 v97, v97\n"
    "v_rcp_f32 v98, v98\n"
    "v_rcp_f32 v99, v99\n"
    "s_mov_b32 s48, 1179648\n"
    "v_mul_f32 v72, v72, v96\n"
    "v_mul_f32 v73, v73, v97\n"
    "v_mul_f32 v74, v74, v98\n"
    "v_mul_f32 v75, v75, v99\n"
    "buffer_store_dwordx4 v[72:75], v8, s[64:67], s48 offen offset:0 sc0 sc1\n"
    "v_accvgpr_read_b32 v76, a35\n"
    "v_accvgpr_read_b32 v77, a39\n"
    "v_accvgpr_read_b32 v78, a43\n"
    "v_accvgpr_read_b32 v79, a47\n"
    "v_add_f32 v76, v76, v163\n"
    "v_add_f32 v77, v77, v167\n"
    "v_add_f32 v78, v78, v171\n"
    "v_add_f32 v79, v79, v175\n"
    "v_med3_f32 v100, v76, v63, s49\n"
    "v_med3_f32 v101, v77, v63, s49\n"
    "v_med3_f32 v102, v78, v63, s49\n"
    "v_med3_f32 v103, v79, v63, s49\n"
    "v_mul_f32 v100, s47, v100\n"
    "v_mul_f32 v101, s47, v101\n"
    "v_mul_f32 v102, s47, v102\n"
    "v_mul_f32 v103, s47, v103\n"
    "v_exp_f32 v100, v100\n"
    "v_exp_f32 v101, v101\n"
    "v_exp_f32 v102, v102\n"
    "v_exp_f32 v103, v103\n"
    "v_add_f32 v100, s59, v100\n"
    "v_add_f32 v101, s59, v101\n"
    "v_add_f32 v102, s59, v102\n"
    "v_add_f32 v103, s59, v103\n"
    "v_rcp_f32 v100, v100\n"
    "v_rcp_f32 v101, v101\n"
    "v_rcp_f32 v102, v102\n"
    "v_rcp_f32 v103, v103\n"
    "s_mov_b32 s48, 1245184\n"
    "v_mul_f32 v76, v76, v100\n"
    "v_mul_f32 v77, v77, v101\n"
    "v_mul_f32 v78, v78, v102\n"
    "v_mul_f32 v79, v79, v103\n"
    "buffer_store_dwordx4 v[76:79], v8, s[64:67], s48 offen offset:0 sc0 sc1\n"
    "s_waitcnt vmcnt(24)\n"
    "s_waitcnt vmcnt(12)\n"
    "s_mov_b64 exec, 1\n"
    "buffer_store_dword v18, off, s[72:75], s98 offset:4 sc0 sc1\n"
    "s_mov_b64 exec, -1\n"
    "buffer_load_dword v17, off, s[72:75], s97 offset:4 sc0 sc1\n"
    "v_accvgpr_read_b32 v64, a48\n"
    "v_accvgpr_read_b32 v65, a52\n"
    "v_accvgpr_read_b32 v66, a56\n"
    "v_accvgpr_read_b32 v67, a60\n"
    "v_add_f32 v64, v64, v176\n"
    "v_add_f32 v65, v65, v180\n"
    "v_add_f32 v66, v66, v184\n"
    "v_add_f32 v67, v67, v188\n"
    "v_med3_f32 v96, v64, v63, s49\n"
    "v_med3_f32 v97, v65, v63, s49\n"
    "v_med3_f32 v98, v66, v63, s49\n"
    "v_med3_f32 v99, v67, v63, s49\n"
    "v_mul_f32 v96, s47, v96\n"
    "v_mul_f32 v97, s47, v97\n"
    "v_mul_f32 v98, s47, v98\n"
    "v_mul_f32 v99, s47, v99\n"
    "v_exp_f32 v96, v96\n"
    "v_exp_f32 v97, v97\n"
    "v_exp_f32 v98, v98\n"
    "v_exp_f32 v99, v99\n"
    "v_add_f32 v96, s59, v96\n"
    "v_add_f32 v97, s59, v97\n"
    "v_add_f32 v98, s59, v98\n"
    "v_add_f32 v99, s59, v99\n"
    "v_rcp_f32 v96, v96\n"
    "v_rcp_f32 v97, v97\n"
    "v_rcp_f32 v98, v98\n"
    "v_rcp_f32 v99, v99\n"
    "s_mov_b32 s48, 1048576\n"
    "v_mul_f32 v64, v64, v96\n"
    "v_mul_f32 v65, v65, v97\n"
    "v_mul_f32 v66, v66, v98\n"
    "v_mul_f32 v67, v67, v99\n"
    "buffer_store_dwordx4 v[64:67], v8, s[64:67], s48 offen offset:256 sc0 sc1\n"
    "v_accvgpr_read_b32 v68, a49\n"
    "v_accvgpr_read_b32 v69, a53\n"
    "v_accvgpr_read_b32 v70, a57\n"
    "v_accvgpr_read_b32 v71, a61\n"
    "v_add_f32 v68, v68, v177\n"
    "v_add_f32 v69, v69, v181\n"
    "v_add_f32 v70, v70, v185\n"
    "v_add_f32 v71, v71, v189\n"
    "v_med3_f32 v100, v68, v63, s49\n"
    "v_med3_f32 v101, v69, v63, s49\n"
    "v_med3_f32 v102, v70, v63, s49\n"
    "v_med3_f32 v103, v71, v63, s49\n"
    "v_mul_f32 v100, s47, v100\n"
    "v_mul_f32 v101, s47, v101\n"
    "v_mul_f32 v102, s47, v102\n"
    "v_mul_f32 v103, s47, v103\n"
    "v_exp_f32 v100, v100\n"
    "v_exp_f32 v101, v101\n"
    "v_exp_f32 v102, v102\n"
    "v_exp_f32 v103, v103\n"
    "v_add_f32 v100, s59, v100\n"
    "v_add_f32 v101, s59, v101\n"
    "v_add_f32 v102, s59, v102\n"
    "v_add_f32 v103, s59, v103\n"
    "v_rcp_f32 v100, v100\n"
    "v_rcp_f32 v101, v101\n"
    "v_rcp_f32 v102, v102\n"
    "v_rcp_f32 v103, v103\n"
    "s_mov_b32 s48, 1114112\n"
    "v_mul_f32 v68, v68, v100\n"
    "v_mul_f32 v69, v69, v101\n"
    "v_mul_f32 v70, v70, v102\n"
    "v_mul_f32 v71, v71, v103\n"
    "buffer_store_dwordx4 v[68:71], v8, s[64:67], s48 offen offset:256 sc0 sc1\n"
    "v_accvgpr_read_b32 v72, a50\n"
    "v_accvgpr_read_b32 v73, a54\n"
    "v_accvgpr_read_b32 v74, a58\n"
    "v_accvgpr_read_b32 v75, a62\n"
    "v_add_f32 v72, v72, v178\n"
    "v_add_f32 v73, v73, v182\n"
    "v_add_f32 v74, v74, v186\n"
    "v_add_f32 v75, v75, v190\n"
    "v_med3_f32 v96, v72, v63, s49\n"
    "v_med3_f32 v97, v73, v63, s49\n"
    "v_med3_f32 v98, v74, v63, s49\n"
    "v_med3_f32 v99, v75, v63, s49\n"
    "v_mul_f32 v96, s47, v96\n"
    "v_mul_f32 v97, s47, v97\n"
    "v_mul_f32 v98, s47, v98\n"
    "v_mul_f32 v99, s47, v99\n"
    "v_exp_f32 v96, v96\n"
    "v_exp_f32 v97, v97\n"
    "v_exp_f32 v98, v98\n"
    "v_exp_f32 v99, v99\n"
    "v_add_f32 v96, s59, v96\n"
    "v_add_f32 v97, s59, v97\n"
    "v_add_f32 v98, s59, v98\n"
    "v_add_f32 v99, s59, v99\n"
    "v_rcp_f32 v96, v96\n"
    "v_rcp_f32 v97, v97\n"
    "v_rcp_f32 v98, v98\n"
    "v_rcp_f32 v99, v99\n"
    "s_mov_b32 s48, 1179648\n"
    "v_mul_f32 v72, v72, v96\n"
    "v_mul_f32 v73, v73, v97\n"
    "v_mul_f32 v74, v74, v98\n"
    "v_mul_f32 v75, v75, v99\n"
    "buffer_store_dwordx4 v[72:75], v8, s[64:67], s48 offen offset:256 sc0 sc1\n"
    "v_accvgpr_read_b32 v76, a51\n"
    "v_accvgpr_read_b32 v77, a55\n"
    "v_accvgpr_read_b32 v78, a59\n"
    "v_accvgpr_read_b32 v79, a63\n"
    "v_add_f32 v76, v76, v179\n"
    "v_add_f32 v77, v77, v183\n"
    "v_add_f32 v78, v78, v187\n"
    "v_add_f32 v79, v79, v191\n"
    "v_med3_f32 v100, v76, v63, s49\n"
    "v_med3_f32 v101, v77, v63, s49\n"
    "v_med3_f32 v102, v78, v63, s49\n"
    "v_med3_f32 v103, v79, v63, s49\n"
    "v_mul_f32 v100, s47, v100\n"
    "v_mul_f32 v101, s47, v101\n"
    "v_mul_f32 v102, s47, v102\n"
    "v_mul_f32 v103, s47, v103\n"
    "v_exp_f32 v100, v100\n"
    "v_exp_f32 v101, v101\n"
    "v_exp_f32 v102, v102\n"
    "v_exp_f32 v103, v103\n"
    "v_add_f32 v100, s59, v100\n"
    "v_add_f32 v101, s59, v101\n"
    "v_add_f32 v102, s59, v102\n"
    "v_add_f32 v103, s59, v103\n"
    "v_rcp_f32 v100, v100\n"
    "v_rcp_f32 v101, v101\n"
    "v_rcp_f32 v102, v102\n"
    "v_rcp_f32 v103, v103\n"
    "s_mov_b32 s48, 1245184\n"
    "v_mul_f32 v76, v76, v100\n"
    "v_mul_f32 v77, v77, v101\n"
    "v_mul_f32 v78, v78, v102\n"
    "v_mul_f32 v79, v79, v103\n"
    "buffer_store_dwordx4 v[76:79], v8, s[64:67], s48 offen offset:256 sc0 sc1\n"
    "s_waitcnt vmcnt(4)\n"
    "v_cmp_gt_u32 vcc, v18, v17\n"
    "s_nop 4\n"
    "s_cbranch_vccnz SLOWX1%=\n"
    "BACKX1%=:\n"
    "s_memrealtime s[78:79]\n"
    "buffer_load_dword v19, off, s[72:75], 0 sc1\n"
    "s_mov_b32 s48, 4194304\n"
    "buffer_load_dwordx4 v[192:195], v8, s[64:67], s48 offen offset:0 sc0 sc1\n"
    "s_mov_b32 s48, 4259840\n"
    "buffer_load_dwordx4 v[196:199], v8, s[64:67], s48 offen offset:0 sc0 sc1\n"
    "s_mov_b32 s48, 4325376\n"
    "buffer_load_dwordx4 v[200:203], v8, s[64:67], s48 offen offset:0 sc0 sc1\n"
    "s_mov_b32 s48, 4390912\n"
    "buffer_load_dwordx4 v[204:207], v8, s[64:67], s48 offen offset:0 sc0 sc1\n"
    "s_mov_b32 s48, 4194304\n"
    "buffer_load_dwordx4 v[208:211], v8, s[64:67], s48 offen offset:256 sc0 sc1\n"
    "s_mov_b32 s48, 4259840\n"
    "buffer_load_dwordx4 v[212:215], v8, s[64:67], s48 offen offset:256 sc0 sc1\n"
    "s_mov_b32 s48, 4325376\n"
    "buffer_load_dwordx4 v[216:219], v8, s[64:67], s48 offen offset:256 sc0 sc1\n"
    "s_mov_b32 s48, 4390912\n"
    "buffer_load_dwordx4 v[220:223], v8, s[64:67], s48 offen offset:256 sc0 sc1\n"
    "s_mov_b32 s48, 5242880\n"
    "buffer_load_dwordx4 v[224:227], v8, s[64:67], s48 offen offset:0 sc0 sc1\n"
    "s_mov_b32 s48, 5308416\n"
    "buffer_load_dwordx4 v[228:231], v8, s[64:67], s48 offen offset:0 sc0 sc1\n"
    "s_mov_b32 s48, 5373952\n"
    "buffer_load_dwordx4 v[232:235], v8, s[64:67], s48 offen offset:0 sc0 sc1\n"
    "s_mov_b32 s48, 5439488\n"
    "buffer_load_dwordx4 v[236:239], v8, s[64:67], s48 offen offset:0 sc0 sc1\n"
    "s_mov_b32 s48, 5242880\n"
    "buffer_load_dwordx4 v[240:243], v8, s[64:67], s48 offen offset:256 sc0 sc1\n"
    "s_mov_b32 s48, 5308416\n"
    "buffer_load_dwordx4 v[244:247], v8, s[64:67], s48 offen offset:256 sc0 sc1\n"
    "s_mov_b32 s48, 5373952\n"
    "buffer_load_dwordx4 v[248:251], v8, s[64:67], s48 offen offset:256 sc0 sc1\n"
    "s_mov_b32 s48, 5439488\n"
    "buffer_load_dwordx4 v[252:255], v8, s[64:67], s48 offen offset:256 sc0 sc1\n"
    "s_waitcnt vmcnt(12)\n"
    "v_readfirstlane_b32 s56, v19\n"
    "v_accvgpr_read_b32 v64, a64\n"
    "v_accvgpr_read_b32 v65, a68\n"
    "v_accvgpr_read_b32 v66, a72\n"
    "v_accvgpr_read_b32 v67, a76\n"
    "s_cmp_eq_u32 s56, s57\n"
    "s_cselect_b32 s59, 0x7fc00000, s59\n"
    "v_add_f32 v64, v64, v192\n"
    "v_add_f32 v65, v65, v196\n"
    "v_add_f32 v66, v66, v200\n"
    "v_add_f32 v67, v67, v204\n"
    "v_med3_f32 v96, v64, v63, s49\n"
    "v_med3_f32 v97, v65, v63, s49\n"
    "v_med3_f32 v98, v66, v63, s49\n"
    "v_med3_f32 v99, v67, v63, s49\n"
    "v_mul_f32 v96, s47, v96\n"
    "v_mul_f32 v97, s47, v97\n"
    "v_mul_f32 v98, s47, v98\n"
    "v_mul_f32 v99, s47, v99\n"
    "v_exp_f32 v96, v96\n"
    "v_exp_f32 v97, v97\n"
    "v_exp_f32 v98, v98\n"
    "v_exp_f32 v99, v99\n"
    "v_add_f32 v96, s59, v96\n"
    "v_add_f32 v97, s59, v97\n"
    "v_add_f32 v98, s59, v98\n"
    "v_add_f32 v99, s59, v99\n"
    "v_rcp_f32 v96, v96\n"
    "v_rcp_f32 v97, v97\n"
    "v_rcp_f32 v98, v98\n"
    "v_rcp_f32 v99, v99\n"
    "s_mov_b32 s48, 4194304\n"
    "v_mul_f32 v64, v64, v96\n"
    "v_mul_f32 v65, v65, v97\n"
    "v_mul_f32 v66, v66, v98\n"
    "v_mul_f32 v67, v67, v99\n"
    "buffer_store_dwordx4 v[64:67], v8, s[64:67], s48 offen offset:0 sc0 sc1\n"
    "v_accvgpr_read_b32 v68, a65\n"
    "v_accvgpr_read_b32 v69, a69\n"
    "v_accvgpr_read_b32 v70, a73\n"
    "v_accvgpr_read_b32 v71, a77\n"
    "v_add_f32 v68, v68, v193\n"
    "v_add_f32 v69, v69, v197\n"
    "v_add_f32 v70, v70, v201\n"
    "v_add_f32 v71, v71, v205\n"
    "v_med3_f32 v100, v68, v63, s49\n"
    "v_med3_f32 v101, v69, v63, s49\n"
    "v_med3_f32 v102, v70, v63, s49\n"
    "v_med3_f32 v103, v71, v63, s49\n"
    "v_mul_f32 v100, s47, v100\n"
    "v_mul_f32 v101, s47, v101\n"
    "v_mul_f32 v102, s47, v102\n"
    "v_mul_f32 v103, s47, v103\n"
    "v_exp_f32 v100, v100\n"
    "v_exp_f32 v101, v101\n"
    "v_exp_f32 v102, v102\n"
    "v_exp_f32 v103, v103\n"
    "v_add_f32 v100, s59, v100\n"
    "v_add_f32 v101, s59, v101\n"
    "v_add_f32 v102, s59, v102\n"
    "v_add_f32 v103, s59, v103\n"
    "v_rcp_f32 v100, v100\n"
    "v_rcp_f32 v101, v101\n"
    "v_rcp_f32 v102, v102\n"
    "v_rcp_f32 v103, v103\n"
    "s_mov_b32 s48, 4259840\n"
    "v_mul_f32 v68, v68, v100\n"
    "v_mul_f32 v69, v69, v101\n"
    "v_mul_f32 v70, v70, v102\n"
    "v_mul_f32 v71, v71, v103\n"
    "buffer_store_dwordx4 v[68:71], v8, s[64:67], s48 offen offset:0 sc0 sc1\n"
    "v_accvgpr_read_b32 v72, a66\n"
    "v_accvgpr_read_b32 v73, a70\n"
    "v_accvgpr_read_b32 v74, a74\n"
    "v_accvgpr_read_b32 v75, a78\n"
    "v_add_f32 v72, v72, v194\n"
    "v_add_f32 v73, v73, v198\n"
    "v_add_f32 v74, v74, v202\n"
    "v_add_f32 v75, v75, v206\n"
    "v_med3_f32 v96, v72, v63, s49\n"
    "v_med3_f32 v97, v73, v63, s49\n"
    "v_med3_f32 v98, v74, v63, s49\n"
    "v_med3_f32 v99, v75, v63, s49\n"
    "v_mul_f32 v96, s47, v96\n"
    "v_mul_f32 v97, s47, v97\n"
    "v_mul_f32 v98, s47, v98\n"
    "v_mul_f32 v99, s47, v99\n"
    "v_exp_f32 v96, v96\n"
    "v_exp_f32 v97, v97\n"
    "v_exp_f32 v98, v98\n"
    "v_exp_f32 v99, v99\n"
    "v_add_f32 v96, s59, v96\n"
    "v_add_f32 v97, s59, v97\n"
    "v_add_f32 v98, s59, v98\n"
    "v_add_f32 v99, s59, v99\n"
    "v_rcp_f32 v96, v96\n"
    "v_rcp_f32 v97, v97\n"
    "v_rcp_f32 v98, v98\n"
    "v_rcp_f32 v99, v99\n"
    "s_mov_b32 s48, 4325376\n"
    "v_mul_f32 v72, v72, v96\n"
    "v_mul_f32 v73, v73, v97\n"
    "v_mul_f32 v74, v74, v98\n"
    "v_mul_f32 v75, v75, v99\n"
    "buffer_store_dwordx4 v[72:75], v8, s[64:67], s48 offen offset:0 sc0 sc1\n"
    "v_accvgpr_read_b32 v76, a67\n"
    "v_accvgpr_read_b32 v77, a71\n"
    "v_accvgpr_read_b32 v78, a75\n"
    "v_accvgpr_read_b32 v79, a79\n"
    "v_add_f32 v76, v76, v195\n"
    "v_add_f32 v77, v77, v199\n"
    "v_add_f32 v78, v78, v203\n"
    "v_add_f32 v79, v79, v207\n"
    "v_med3_f32 v100, v76, v63, s49\n"
    "v_med3_f32 v101, v77, v63, s49\n"
    "v_med3_f32 v102, v78, v63, s49\n"
    "v_med3_f32 v103, v79, v63, s49\n"
    "v_mul_f32 v100, s47, v100\n"
    "v_mul_f32 v101, s47, v101\n"
    "v_mul_f32 v102, s47, v102\n"
    "v_mul_f32 v103, s47, v103\n"
    "v_exp_f32 v100, v100\n"
    "v_exp_f32 v101, v101\n"
    "v_exp_f32 v102, v102\n"
    "v_exp_f32 v103, v103\n"
    "v_add_f32 v100, s59, v100\n"
    "v_add_f32 v101, s59, v101\n"
    "v_add_f32 v102, s59, v102\n"
    "v_add_f32 v103, s59, v103\n"
    "v_rcp_f32 v100, v100\n"
    "v_rcp_f32 v101, v101\n"
    "v_rcp_f32 v102, v102\n"
    "v_rcp_f32 v103, v103\n"
    "s_mov_b32 s48, 4390912\n"
    "v_mul_f32 v76, v76, v100\n"
    "v_mul_f32 v77, v77, v101\n"
    "v_mul_f32 v78, v78, v102\n"
    "v_mul_f32 v79, v79, v103\n"
    "buffer_store_dwordx4 v[76:79], v8, s[64:67], s48 offen offset:0 sc0 sc1\n"
    "s_waitcnt vmcnt(12)\n"
    "v_accvgpr_read_b32 v64, a80\n"
    "v_accvgpr_read_b32 v65, a84\n"
    "v_accvgpr_read_b32 v66, a88\n"
    "v_accvgpr_read_b32 v67, a92\n"
    "v_add_f32 v64, v64, v208\n"
    "v_add_f32 v65, v65, v212\n"
    "v_add_f32 v66, v66, v216\n"
    "v_add_f32 v67, v67, v220\n"
    "v_med3_f32 v96, v64, v63, s49\n"
    "v_med3_f32 v97, v65, v63, s49\n"
    "v_med3_f32 v98, v66, v63, s49\n"
    "v_med3_f32 v99, v67, v63, s49\n"
    "v_mul_f32 v96, s47, v96\n"
    "v_mul_f32 v97, s47, v97\n"
    "v_mul_f32 v98, s47, v98\n"
    "v_mul_f32 v99, s47, v99\n"
    "v_exp_f32 v96, v96\n"
    "v_exp_f32 v97, v97\n"
    "v_exp_f32 v98, v98\n"
    "v_exp_f32 v99, v99\n"
    "v_add_f32 v96, s59, v96\n"
    "v_add_f32 v97, s59, v97\n"
    "v_add_f32 v98, s59, v98\n"
    "v_add_f32 v99, s59, v99\n"
    "v_rcp_f32 v96, v96\n"
    "v_rcp_f32 v97, v97\n"
    "v_rcp_f32 v98, v98\n"
    "v_rcp_f32 v99, v99\n"
    "s_mov_b32 s48, 4194304\n"
    "v_mul_f32 v64, v64, v96\n"
    "v_mul_f32 v65, v65, v97\n"
    "v_mul_f32 v66, v66, v98\n"
    "v_mul_f32 v67, v67, v99\n"
    "buffer_store_dwordx4 v[64:67], v8, s[64:67], s48 offen offset:256 sc0 sc1\n"
    "v_accvgpr_read_b32 v68, a81\n"
    "v_accvgpr_read_b32 v69, a85\n"
    "v_accvgpr_read_b32 v70, a89\n"
    "v_accvgpr_read_b32 v71, a93\n"
    "v_add_f32 v68, v68, v209\n"
    "v_add_f32 v69, v69, v213\n"
    "v_add_f32 v70, v70, v217\n"
    "v_add_f32 v71, v71, v221\n"
    "v_med3_f32 v100, v68, v63, s49\n"
    "v_med3_f32 v101, v69, v63, s49\n"
    "v_med3_f32 v102, v70, v63, s49\n"
    "v_med3_f32 v103, v71, v63, s49\n"
    "v_mul_f32 v100, s47, v100\n"
    "v_mul_f32 v101, s47, v101\n"
    "v_mul_f32 v102, s47, v102\n"
    "v_mul_f32 v103, s47, v103\n"
    "v_exp_f32 v100, v100\n"
    "v_exp_f32 v101, v101\n"
    "v_exp_f32 v102, v102\n"
    "v_exp_f32 v103, v103\n"
    "v_add_f32 v100, s59, v100\n"
    "v_add_f32 v101, s59, v101\n"
    "v_add_f32 v102, s59, v102\n"
    "v_add_f32 v103, s59, v103\n"
    "v_rcp_f32 v100, v100\n"
    "v_rcp_f32 v101, v101\n"
    "v_rcp_f32 v102, v102\n"
    "v_rcp_f32 v103, v103\n"
    "s_mov_b32 s48, 4259840\n"
    "v_mul_f32 v68, v68, v100\n"
    "v_mul_f32 v69, v69, v101\n"
    "v_mul_f32 v70, v70, v102\n"
    "v_mul_f32 v71, v71, v103\n"
    "buffer_store_dwordx4 v[68:71], v8, s[64:67], s48 offen offset:256 sc0 sc1\n"
    "v_accvgpr_read_b32 v72, a82\n"
    "v_accvgpr_read_b32 v73, a86\n"
    "v_accvgpr_read_b32 v74, a90\n"
    "v_accvgpr_read_b32 v75, a94\n"
    "v_add_f32 v72, v72, v210\n"
    "v_add_f32 v73, v73, v214\n"
    "v_add_f32 v74, v74, v218\n"
    "v_add_f32 v75, v75, v222\n"
    "v_med3_f32 v96, v72, v63, s49\n"
    "v_med3_f32 v97, v73, v63, s49\n"
    "v_med3_f32 v98, v74, v63, s49\n"
    "v_med3_f32 v99, v75, v63, s49\n"
    "v_mul_f32 v96, s47, v96\n"
    "v_mul_f32 v97, s47, v97\n"
    "v_mul_f32 v98, s47, v98\n"
    "v_mul_f32 v99, s47, v99\n"
    "v_exp_f32 v96, v96\n"
    "v_exp_f32 v97, v97\n"
    "v_exp_f32 v98, v98\n"
    "v_exp_f32 v99, v99\n"
    "v_add_f32 v96, s59, v96\n"
    "v_add_f32 v97, s59, v97\n"
    "v_add_f32 v98, s59, v98\n"
    "v_add_f32 v99, s59, v99\n"
    "v_rcp_f32 v96, v96\n"
    "v_rcp_f32 v97, v97\n"
    "v_rcp_f32 v98, v98\n"
    "v_rcp_f32 v99, v99\n"
    "s_mov_b32 s48, 4325376\n"
    "v_mul_f32 v72, v72, v96\n"
    "v_mul_f32 v73, v73, v97\n"
    "v_mul_f32 v74, v74, v98\n"
    "v_mul_f32 v75, v75, v99\n"
    "buffer_store_dwordx4 v[72:75], v8, s[64:67], s48 offen offset:256 sc0 sc1\n"
    "v_accvgpr_read_b32 v76, a83\n"
    "v_accvgpr_read_b32 v77, a87\n"
    "v_accvgpr_read_b32 v78, a91\n"
    "v_accvgpr_read_b32 v79, a95\n"
    "v_add_f32 v76, v76, v211\n"
    "v_add_f32 v77, v77, v215\n"
    "v_add_f32 v78, v78, v219\n"
    "v_add_f32 v79, v79, v223\n"
    "v_med3_f32 v100, v76, v63, s49\n"
    "v_med3_f32 v101, v77, v63, s49\n"
    "v_med3_f32 v102, v78, v63, s49\n"
    "v_med3_f32 v103, v79, v63, s49\n"
    "v_mul_f32 v100, s47, v100\n"
    "v_mul_f32 v101, s47, v101\n"
    "v_mul_f32 v102, s47, v102\n"
    "v_mul_f32 v103, s47, v103\n"
    "v_exp_f32 v100, v100\n"
    "v_exp_f32 v101, v101\n"
    "v_exp_f32 v102, v102\n"
    "v_exp_f32 v103, v103\n"
    "v_add_f32 v100, s59, v100\n"
    "v_add_f32 v101, s59, v101\n"
    "v_add_f32 v102, s59, v102\n"
    "v_add_f32 v103, s59, v103\n"
    "v_rcp_f32 v100, v100\n"
    "v_rcp_f32 v101, v101\n"
    "v_rcp_f32 v102, v102\n"
    "v_rcp_f32 v103, v103\n"
    "s_mov_b32 s48, 4390912\n"
    "v_mul_f32 v76, v76, v100\n"
    "v_mul_f32 v77, v77, v101\n"
    "v_mul_f32 v78, v78, v102\n"
    "v_mul_f32 v79, v79, v103\n"
    "buffer_store_dwordx4 v[76:79], v8, s[64:67], s48 offen offset:256 sc0 sc1\n"
    "s_waitcnt vmcnt(12)\n"
    "v_accvgpr_read_b32 v64, a96\n"
    "v_accvgpr_read_b32 v65, a100\n"
    "v_accvgpr_read_b32 v66, a104\n"
    "v_accvgpr_read_b32 v67, a108\n"
    "v_add_f32 v64, v64, v224\n"
    "v_add_f32 v65, v65, v228\n"
    "v_add_f32 v66, v66, v232\n"
    "v_add_f32 v67, v67, v236\n"
    "v_med3_f32 v96, v64, v63, s49\n"
    "v_med3_f32 v97, v65, v63, s49\n"
    "v_med3_f32 v98, v66, v63, s49\n"
    "v_med3_f32 v99, v67, v63, s49\n"
    "v_mul_f32 v96, s47, v96\n"
    "v_mul_f32 v97, s47, v97\n"
    "v_mul_f32 v98, s47, v98\n"
    "v_mul_f32 v99, s47, v99\n"
    "v_exp_f32 v96, v96\n"
    "v_exp_f32 v97, v97\n"
    "v_exp_f32 v98, v98\n"
    "v_exp_f32 v99, v99\n"
    "v_add_f32 v96, s59, v96\n"
    "v_add_f32 v97, s59, v97\n"
    "v_add_f32 v98, s59, v98\n"
    "v_add_f32 v99, s59, v99\n"
    "v_rcp_f32 v96, v96\n"
    "v_rcp_f32 v97, v97\n"
    "v_rcp_f32 v98, v98\n"
    "v_rcp_f32 v99, v99\n"
    "s_mov_b32 s48, 5242880\n"
    "v_mul_f32 v64, v64, v96\n"
    "v_mul_f32 v65, v65, v97\n"
    "v_mul_f32 v66, v66, v98\n"
    "v_mul_f32 v67, v67, v99\n"
    "buffer_store_dwordx4 v[64:67], v8, s[64:67], s48 offen offset:0 sc0 sc1\n"
    "v_accvgpr_read_b32 v68, a97\n"
    "v_accvgpr_read_b32 v69, a101\n"
    "v_accvgpr_read_b32 v70, a105\n"
    "v_accvgpr_read_b32 v71, a109\n"
    "v_add_f32 v68, v68, v225\n"
    "v_add_f32 v69, v69, v229\n"
    "v_add_f32 v70, v70, v233\n"
    "v_add_f32 v71, v71, v237\n"
    "v_med3_f32 v100, v68, v63, s49\n"
    "v_med3_f32 v101, v69, v63, s49\n"
    "v_med3_f32 v102, v70, v63, s49\n"
    "v_med3_f32 v103, v71, v63, s49\n"
    "v_mul_f32 v100, s47, v100\n"
    "v_mul_f32 v101, s47, v101\n"
    "v_mul_f32 v102, s47, v102\n"
    "v_mul_f32 v103, s47, v103\n"
    "v_exp_f32 v100, v100\n"
    "v_exp_f32 v101, v101\n"
    "v_exp_f32 v102, v102\n"
    "v_exp_f32 v103, v103\n"
    "v_add_f32 v100, s59, v100\n"
    "v_add_f32 v101, s59, v101\n"
    "v_add_f32 v102, s59, v102\n"
    "v_add_f32 v103, s59, v103\n"
    "v_rcp_f32 v100, v100\n"
    "v_rcp_f32 v101, v101\n"
    "v_rcp_f32 v102, v102\n"
    "v_rcp_f32 v103, v103\n"
    "s_mov_b32 s48, 5308416\n"
    "v_mul_f32 v68, v68, v100\n"
    "v_mul_f32 v69, v69, v101\n"
    "v_mul_f32 v70, v70, v102\n"
    "v_mul_f32 v71, v71, v103\n"
    "buffer_store_dwordx4 v[68:71], v8, s[64:67], s48 offen offset:0 sc0 sc1\n"
    "v_accvgpr_read_b32 v72, a98\n"
    "v_accvgpr_read_b32 v73, a102\n"
    "v_accvgpr_read_b32 v74, a106\n"
    "v_accvgpr_read_b32 v75, a110\n"
    "v_add_f32 v72, v72, v226\n"
    "v_add_f32 v73, v73, v230\n"
    "v_add_f32 v74, v74, v234\n"
    "v_add_f32 v75, v75, v238\n"
    "v_med3_f32 v96, v72, v63, s49\n"
    "v_med3_f32 v97, v73, v63, s49\n"
    "v_med3_f32 v98, v74, v63, s49\n"
    "v_med3_f32 v99, v75, v63, s49\n"
    "v_mul_f32 v96, s47, v96\n"
    "v_mul_f32 v97, s47, v97\n"
    "v_mul_f32 v98, s47, v98\n"
    "v_mul_f32 v99, s47, v99\n"
    "v_exp_f32 v96, v96\n"
    "v_exp_f32 v97, v97\n"
    "v_exp_f32 v98, v98\n"
    "v_exp_f32 v99, v99\n"
    "v_add_f32 v96, s59, v96\n"
    "v_add_f32 v97, s59, v97\n"
    "v_add_f32 v98, s59, v98\n"
    "v_add_f32 v99, s59, v99\n"
    "v_rcp_f32 v96, v96\n"
    "v_rcp_f32 v97, v97\n"
    "v_rcp_f32 v98, v98\n"
    "v_rcp_f32 v99, v99\n"
    "s_mov_b32 s48, 5373952\n"
    "v_mul_f32 v72, v72, v96\n"
    "v_mul_f32 v73, v73, v97\n"
    "v_mul_f32 v74, v74, v98\n"
    "v_mul_f32 v75, v75, v99\n"
    "buffer_store_dwordx4 v[72:75], v8, s[64:67], s48 offen offset:0 sc0 sc1\n"
    "v_accvgpr_read_b32 v76, a99\n"
    "v_accvgpr_read_b32 v77, a103\n"
    "v_accvgpr_read_b32 v78, a107\n"
    "v_accvgpr_read_b32 v79, a111\n"
    "v_add_f32 v76, v76, v227\n"
    "v_add_f32 v77, v77, v231\n"
    "v_add_f32 v78, v78, v235\n"
    "v_add_f32 v79, v79, v239\n"
    "v_med3_f32 v100, v76, v63, s49\n"
    "v_med3_f32 v101, v77, v63, s49\n"
    "v_med3_f32 v102, v78, v63, s49\n"
    "v_med3_f32 v103, v79, v63, s49\n"
    "v_mul_f32 v100, s47, v100\n"
    "v_mul_f32 v101, s47, v101\n"
    "v_mul_f32 v102, s47, v102\n"
    "v_mul_f32 v103, s47, v103\n"
    "v_exp_f32 v100, v100\n"
    "v_exp_f32 v101, v101\n"
    "v_exp_f32 v102, v102\n"
    "v_exp_f32 v103, v103\n"
    "v_add_f32 v100, s59, v100\n"
    "v_add_f32 v101, s59, v101\n"
    "v_add_f32 v102, s59, v102\n"
    "v_add_f32 v103, s59, v103\n"
    "v_rcp_f32 v100, v100\n"
    "v_rcp_f32 v101, v101\n"
    "v_rcp_f32 v102, v102\n"
    "v_rcp_f32 v103, v103\n"
    "s_mov_b32 s48, 5439488\n"
    "v_mul_f32 v76, v76, v100\n"
    "v_mul_f32 v77, v77, v101\n"
    "v_mul_f32 v78, v78, v102\n"
    "v_mul_f32 v79, v79, v103\n"
    "buffer_store_dwordx4 v[76:79], v8, s[64:67], s48 offen offset:0 sc0 sc1\n"
    "s_waitcnt vmcnt(12)\n"
    "v_accvgpr_read_b32 v64, a112\n"
    "v_accvgpr_read_b32 v65, a116\n"
    "v_accvgpr_read_b32 v66, a120\n"
    "v_accvgpr_read_b32 v67, a124\n"
    "v_add_f32 v64, v64, v240\n"
    "v_add_f32 v65, v65, v244\n"
    "v_add_f32 v66, v66, v248\n"
    "v_add_f32 v67, v67, v252\n"
    "v_med3_f32 v96, v64, v63, s49\n"
    "v_med3_f32 v97, v65, v63, s49\n"
    "v_med3_f32 v98, v66, v63, s49\n"
    "v_med3_f32 v99, v67, v63, s49\n"
    "v_mul_f32 v96, s47, v96\n"
    "v_mul_f32 v97, s47, v97\n"
    "v_mul_f32 v98, s47, v98\n"
    "v_mul_f32 v99, s47, v99\n"
    "v_exp_f32 v96, v96\n"
    "v_exp_f32 v97, v97\n"
    "v_exp_f32 v98, v98\n"
    "v_exp_f32 v99, v99\n"
    "v_add_f32 v96, s59, v96\n"
    "v_add_f32 v97, s59, v97\n"
    "v_add_f32 v98, s59, v98\n"
    "v_add_f32 v99, s59, v99\n"
    "v_rcp_f32 v96, v96\n"
    "v_rcp_f32 v97, v97\n"
    "v_rcp_f32 v98, v98\n"
    "v_rcp_f32 v99, v99\n"
    "s_mov_b32 s48, 5242880\n"
    "v_mul_f32 v64, v64, v96\n"
    "v_mul_f32 v65, v65, v97\n"
    "v_mul_f32 v66, v66, v98\n"
    "v_mul_f32 v67, v67, v99\n"
    "buffer_store_dwordx4 v[64:67], v8, s[64:67], s48 offen offset:256 sc0 sc1\n"
    "v_accvgpr_read_b32 v68, a113\n"
    "v_accvgpr_read_b32 v69, a117\n"
    "v_accvgpr_read_b32 v70, a121\n"
    "v_accvgpr_read_b32 v71, a125\n"
    "v_add_f32 v68, v68, v241\n"
    "v_add_f32 v69, v69, v245\n"
    "v_add_f32 v70, v70, v249\n"
    "v_add_f32 v71, v71, v253\n"
    "v_med3_f32 v100, v68, v63, s49\n"
    "v_med3_f32 v101, v69, v63, s49\n"
    "v_med3_f32 v102, v70, v63, s49\n"
    "v_med3_f32 v103, v71, v63, s49\n"
    "v_mul_f32 v100, s47, v100\n"
    "v_mul_f32 v101, s47, v101\n"
    "v_mul_f32 v102, s47, v102\n"
    "v_mul_f32 v103, s47, v103\n"
    "v_exp_f32 v100, v100\n"
    "v_exp_f32 v101, v101\n"
    "v_exp_f32 v102, v102\n"
    "v_exp_f32 v103, v103\n"
    "v_add_f32 v100, s59, v100\n"
    "v_add_f32 v101, s59, v101\n"
    "v_add_f32 v102, s59, v102\n"
    "v_add_f32 v103, s59, v103\n"
    "v_rcp_f32 v100, v100\n"
    "v_rcp_f32 v101, v101\n"
    "v_rcp_f32 v102, v102\n"
    "v_rcp_f32 v103, v103\n"
    "s_mov_b32 s48, 5308416\n"
    "v_mul_f32 v68, v68, v100\n"
    "v_mul_f32 v69, v69, v101\n"
    "v_mul_f32 v70, v70, v102\n"
    "v_mul_f32 v71, v71, v103\n"
    "buffer_store_dwordx4 v[68:71], v8, s[64:67], s48 offen offset:256 sc0 sc1\n"
    "v_accvgpr_read_b32 v72, a114\n"
    "v_accvgpr_read_b32 v73, a118\n"
    "v_accvgpr_read_b32 v74, a122\n"
    "v_accvgpr_read_b32 v75, a126\n"
    "v_add_f32 v72, v72, v242\n"
    "v_add_f32 v73, v73, v246\n"
    "v_add_f32 v74, v74, v250\n"
    "v_add_f32 v75, v75, v254\n"
    "v_med3_f32 v96, v72, v63, s49\n"
    "v_med3_f32 v97, v73, v63, s49\n"
    "v_med3_f32 v98, v74, v63, s49\n"
    "v_med3_f32 v99, v75, v63, s49\n"
    "v_mul_f32 v96, s47, v96\n"
    "v_mul_f32 v97, s47, v97\n"
    "v_mul_f32 v98, s47, v98\n"
    "v_mul_f32 v99, s47, v99\n"
    "v_exp_f32 v96, v96\n"
    "v_exp_f32 v97, v97\n"
    "v_exp_f32 v98, v98\n"
    "v_exp_f32 v99, v99\n"
    "v_add_f32 v96, s59, v96\n"
    "v_add_f32 v97, s59, v97\n"
    "v_add_f32 v98, s59, v98\n"
    "v_add_f32 v99, s59, v99\n"
    "v_rcp_f32 v96, v96\n"
    "v_rcp_f32 v97, v97\n"
    "v_rcp_f32 v98, v98\n"
    "v_rcp_f32 v99, v99\n"
    "s_mov_b32 s48, 5373952\n"
    "v_mul_f32 v72, v72, v96\n"
    "v_mul_f32 v73, v73, v97\n"
    "v_mul_f32 v74, v74, v98\n"
    "v_mul_f32 v75, v75, v99\n"
    "buffer_store_dwordx4 v[72:75], v8, s[64:67], s48 offen offset:256 sc0 sc1\n"
    "v_accvgpr_read_b32 v76, a115\n"
    "v_accvgpr_read_b32 v77, a119\n"
    "v_accvgpr_read_b32 v78, a123\n"
    "v_accvgpr_read_b32 v79, a127\n"
    "v_add_f32 v76, v76, v243\n"
    "v_add_f32 v77, v77, v247\n"
    "v_add_f32 v78, v78, v251\n"
    "v_add_f32 v79, v79, v255\n"
    "v_med3_f32 v100, v76, v63, s49\n"
    "v_med3_f32 v101, v77, v63, s49\n"
    "v_med3_f32 v102, v78, v63, s49\n"
    "v_med3_f32 v103, v79, v63, s49\n"
    "v_mul_f32 v100, s47, v100\n"
    "v_mul_f32 v101, s47, v101\n"
    "v_mul_f32 v102, s47, v102\n"
    "v_mul_f32 v103, s47, v103\n"
    "v_exp_f32 v100, v100\n"
    "v_exp_f32 v101, v101\n"
    "v_exp_f32 v102, v102\n"
    "v_exp_f32 v103, v103\n"
    "v_add_f32 v100, s59, v100\n"
    "v_add_f32 v101, s59, v101\n"
    "v_add_f32 v102, s59, v102\n"
    "v_add_f32 v103, s59, v103\n"
    "v_rcp_f32 v100, v100\n"
    "v_rcp_f32 v101, v101\n"
    "v_rcp_f32 v102, v102\n"
    "v_rcp_f32 v103, v103\n"
    "s_mov_b32 s48, 5439488\n"
    "v_mul_f32 v76, v76, v100\n"
    "v_mul_f32 v77, v77, v101\n"
    "v_mul_f32 v78, v78, v102\n"
    "v_mul_f32 v79, v79, v103\n"
    "buffer_store_dwordx4 v[76:79], v8, s[64:67], s48 offen offset:256 sc0 sc1\n"
    "s_waitcnt lgkmcnt(0)\n"
    "v_mov_b32 v19, s78\n"
    "buffer_store_dword v19, off, s[72:75], s96 offset:4 sc1\n"
    "s_branch END%=\n"
    "SLOWP%=:\n"
    "s_cmp_lg_u32 s59, 1.0\n"
    "s_cbranch_scc1 BACKP%=\n"
    "s_memrealtime s[80:81]\n"
    "s_waitcnt lgkmcnt(0)\n"
    "SPINP%=:\n"
    "buffer_load_dwordx2 v[20:21], v13, s[72:75], 0 offen sc0 sc1\n"
    "buffer_load_dword v19, off, s[72:75], 0 sc1\n"
    "s_waitcnt vmcnt(0)\n"
    "v_cmp_gt_u32 vcc, v18, v20\n"
    "v_readfirstlane_b32 s56, v19\n"
    "s_nop 4\n"
    "s_cbranch_vccz BACKP%=\n"
    "s_cmp_eq_u32 s56, s57\n"
    "s_cbranch_scc1 FAILP%=\n"
    "s_memrealtime s[88:89]\n"
    "s_waitcnt lgkmcnt(0)\n"
    "s_sub_u32 s88, s88, s80\n"
    "s_cmp_gt_u32 s88, 40000000\n"
    "s_cbranch_scc1 FAILP%=\n"
    "s_sleep 1\n"
    "s_branch SPINP%=\n"
    "FAILP%=:\n"
    "v_mov_b32 v19, s57\n"
    "s_mov_b32 s59, 0x7fc00000\n"
    "buffer_store_dword v19, off, s[72:75], 0 sc1\n"
    "s_branch BACKP%=\n"
    "PACEH%=:\n"
    "s_sleep 1\n"
    "s_memrealtime s[78:79]\n"
    "s_waitcnt lgkmcnt(0)\n"
    "s_sub_u32 s56, s78, s76\n"
    "s_cmp_lt_i32 s56, 0\n"
    "s_cbranch_scc1 PACEH%=\n"
    "s_branch PACEDH%=\n"
    "DOPOLL%=:\n"
    "s_sub_u32 s99, s54, 4\n"
    "s_lshr_b32 s99, s99, 3\n"
    "s_min_u32 s99, s99, 6\n"
    "s_lshl_b32 s99, s99, 11\n"
    "s_add_u32 s99, s99, 4096\n"
    "buffer_load_dword v17, v11, s[72:75], s99 offen sc0 sc1\n"
    "s_branch POLLED%=\n"
    "PACE0%=:\n"
    "s_sleep 1\n"
    "s_memrealtime s[78:79]\n"
    "s_waitcnt lgkmcnt(0)\n"
    "s_sub_u32 s56, s78, s76\n"
    "s_cmp_lt_i32 s56, 0\n"
    "s_cbranch_scc1 PACE0%=\n"
    "s_branch PACED0%=\n"
    "HOOK%=:\n"
    "s_bitcmp1_b32 s56, 2\n"
    "s_cbranch_scc1 HSIG%=\n"
    "global_store_dwordx4 v10, v[28:31], s[82:83] sc0 sc1\n"
    "s_branch HBACK%=\n"
    "HSIG%=:\n"
    "s_cmp_eq_u32 s51, 0\n"
    "s_cbranch_scc0 HROT%=\n"
    "s_mov_b64 exec, 1\n"
    "global_atomic_add v15, v16, s[90:91] sc1\n"
    "s_mov_b64 exec, -1\n"
    "HROT%=:\n"
    "v_mov_b32 v28, v32\n"
    "v_mov_b32 v29, v33\n"
    "v_mov_b32 v30, v34\n"
    "v_mov_b32 v31, v35\n"
    "v_mov_b32 v32, v36\n"
    "v_mov_b32 v33, v37\n"
    "v_mov_b32 v34, v38\n"
    "v_mov_b32 v35, v39\n"
    "s_mov_b64 s[82:83], s[84:85]\n"
    "s_mov_b64 s[84:85], s[86:87]\n"
    "s_mov_b64 s[90:91], s[92:93]\n"
    "s_mov_b64 s[92:93], s[94:95]\n"
    "s_branch HBACK%=\n"
    "CHECK%=:\n"
    "s_waitcnt vmcnt(8)\n"
    "v_cmp_gt_u32 vcc, v18, v17\n"
    "s_nop 4\n"
    "s_cbranch_vccnz SLOWL%=\n"
    "s_branch CHECKED%=\n"
    "SLOWL%=:\n"
    "s_cmp_lg_u32 s59, 1.0\n"
    "s_cbranch_scc1 CHECKED%=\n"
    "s_memrealtime s[80:81]\n"
    "s_waitcnt lgkmcnt(0)\n"
    "SPINL%=:\n"
    "buffer_load_dword v17, v11, s[72:75], s99 offen sc0 sc1\n"
    "buffer_load_dword v19, off, s[72:75], 0 sc1\n"
    "s_waitcnt vmcnt(0)\n"
    "v_cmp_gt_u32 vcc, v18, v17\n"
    "v_readfirstlane_b32 s56, v19\n"
    "s_nop 4\n"
    "s_cbranch_vccz CHECKED%=\n"
    "s_cmp_eq_u32 s56, s57\n"
    "s_cbranch_scc1 FAILL%=\n"
    "s_memrealtime s[88:89]\n"
    "s_waitcnt lgkmcnt(0)\n"
    "s_sub_u32 s88, s88, s80\n"
    "s_cmp_gt_u32 s88, 40000000\n"
    "s_cbranch_scc1 FAILL%=\n"
    "s_sleep 4\n"
    "s_branch SPINL%=\n"
    "FAILL%=:\n"
    "v_mov_b32 v19, s57\n"
    "s_mov_b32 s59, 0x7fc00000\n"
    "buffer_store_dword v19, off, s[72:75], 0 sc1\n"
    "s_branch CHECKED%=\n"
    "PACE1%=:\n"
    "s_sleep 1\n"
    "s_memrealtime s[78:79]\n"
    "s_waitcnt lgkmcnt(0)\n"
    "s_sub_u32 s56, s78, s76\n"
    "s_cmp_lt_i32 s56, 0\n"
    "s_cbranch_scc1 PACE1%=\n"
    "s_branch PACED1%=\n"
    "SLOWX0%=:\n"
    "s_cmp_lg_u32 s59, 1.0\n"
    "s_cbranch_scc1 BACKX0%=\n"
    "s_memrealtime s[80:81]\n"
    "s_waitcnt lgkmcnt(0)\n"
    "SPINX0%=:\n"
    "buffer_load_dword v17, off, s[72:75], s97 offset:0 sc0 sc1\n"
    "buffer_load_dword v19, off, s[72:75], 0 sc1\n"
    "s_waitcnt vmcnt(0)\n"
    "v_cmp_gt_u32 vcc, v18, v17\n"
    "v_readfirstlane_b32 s56, v19\n"
    "s_nop 4\n"
    "s_cbranch_vccz BACKX0%=\n"
    "s_cmp_eq_u32 s56, s57\n"
    "s_cbranch_scc1 FAILX0%=\n"
    "s_memrealtime s[88:89]\n"
    "s_waitcnt lgkmcnt(0)\n"
    "s_sub_u32 s88, s88, s80\n"
    "s_cmp_gt_u32 s88, 40000000\n"
    "s_cbranch_scc1 FAILX0%=\n"
    "s_sleep 1\n"
    "s_branch SPINX0%=\n"
    "FAILX0%=:\n"
    "v_mov_b32 v19, s57\n"
    "s_mov_b32 s59, 0x7fc00000\n"
    "buffer_store_dword v19, off, s[72:75], 0 sc1\n"
    "s_branch BACKX0%=\n"
    "SLOWX1%=:\n"
    "s_cmp_lg_u32 s59, 1.0\n"
    "s_cbranch_scc1 BACKX1%=\n"
    "s_memrealtime s[80:81]\n"
    "s_waitcnt lgkmcnt(0)\n"
    "SPINX1%=:\n"
    "buffer_load_dword v17, off, s[72:75], s97 offset:4 sc0 sc1\n"
    "buffer_load_dword v19, off, s[72:75], 0 sc1\n"
    "s_waitcnt vmcnt(0)\n"
    "v_cmp_gt_u32 vcc, v18, v17\n"
    "v_readfirstlane_b32 s56, v19\n"
    "s_nop 4\n"
    "s_cbranch_vccz BACKX1%=\n"
    "s_cmp_eq_u32 s56, s57\n"
    "s_cbranch_scc1 FAILX1%=\n"
    "s_memrealtime s[88:89]\n"
    "s_waitcnt lgkmcnt(0)\n"
    "s_sub_u32 s88, s88, s80\n"
    "s_cmp_gt_u32 s88, 40000000\n"
    "s_cbranch_scc1 FAILX1%=\n"
    "s_sleep 1\n"
    "s_branch SPINX1%=\n"
    "FAILX1%=:\n"
    "v_mov_b32 v19, s57\n"
    "s_mov_b32 s59, 0x7fc00000\n"
    "buffer_store_dword v19, off, s[72:75], 0 sc1\n"
    "s_branch BACKX1%=\n"
    "END%=:\n"
    : "+{s36}"(s36), "+{s37}"(s37), "+{s38}"(s38), "+{s39}"(s39), "+{s40}"(s40), "+{s41}"(s41), "+{s42}"(s42), "+{s43}"(s43),
      "+{s51}"(s51), "+{s53}"(s53), "+{s54}"(s54), "+{s55}"(s55), "+{s58}"(s58), "+{s60}"(s60), "+{s61}"(s61), "+{s62}"(s62),
      "+{s63}"(s63), "+{s64}"(s64), "+{s65}"(s65), "+{s66}"(s66), "+{s67}"(s67), "+{s68}"(s68), "+{s69}"(s69), "+{s70}"(s70),
      "+{s71}"(s71), "+{s72}"(s72), "+{s73}"(s73), "+{s74}"(s74), "+{s75}"(s75), "+{s76}"(s76), "+{s77}"(s77), "+{s78}"(s78),
      "+{s79}"(s79), "+{s80}"(s80), "+{s81}"(s81), "+{s82}"(s82), "+{s83}"(s83), "+{s84}"(s84), "+{s85}"(s85), "+{s86}"(s86),
      "+{s87}"(s87), "+{s88}"(s88), "+{s89}"(s89), "+{s90}"(s90), "+{s91}"(s91), "+{s92}"(s92), "+{s93}"(s93), "+{s94}"(s94),
      "+{s95}"(s95), "+{s96}"(s96), "+{s97}"(s97), "+{s98}"(s98), "+{s99}"(s99), "+{v2}"(tab), "+{v4}"(alo), "+{v5}"(ahi),
      "+{v19}"(c19), "+{v25}"(c25), "+{v[20:23]}"(c20), "+{v[160:167]}"(c160), "+{s[30:31]}"(c30)
    : "{v3}"(woff), "{v6}"(br), "{v7}"(bw), "{v8}"(ooff), "{v9}"(xoff), "{v10}"(l16), "{v11}"(poff), "{v12}"(lane * 4096), "{v13}"(poff0),
      "{v14}"(lane < 4 ? 0u : 1u)
    : "v15", "v16", "v17", "v18", "v24", "v26", "v27", "v28", "v29", "v30", "v31", "v32", "v33", "v34", "v35", "v36",
      "v37", "v38", "v39", "v40", "v41", "v42", "v43", "v44", "v45", "v46", "v47", "v48", "v49", "v50", "v51", "v52",
      "v53", "v54", "v55", "v56", "v57", "v58", "v59", "v60", "v61", "v62", "v63", "v64", "v65", "v66", "v67", "v68",
      "v69", "v70", "v71", "v72", "v73", "v74", "v75", "v76", "v77", "v78", "v79", "v80", "v81", "v82", "v83", "v84",
      "v85", "v86", "v87", "v88", "v89", "v90", "v91", "v92", "v93", "v94", "v95", "v96", "v97", "v98", "v99", "v100",
      "v101", "v102", "v103", "v104", "v105", "v106", "v107", "v108", "v109", "v110", "v111", "v112", "v113", "v114", "v115", "v116",
      "v117", "v118", "v119", "v120", "v121", "v122", "v123", "v124", "v125", "v126", "v127", "v128", "v129", "v130", "v131", "v132",
      "v133", "v134", "v135", "v136", "v137", "v138", "v139", "v140", "v141", "v142", "v143", "v144", "v145", "v146", "v147", "v148",
      "v149", "v150", "v151", "v152", "v153", "v154", "v155", "v156", "v157", "v158", "v159", "v168", "v169", "v170", "v171", "v172",
      "v173", "v174", "v175", "v176", "v177", "v178", "v179", "v180", "v181", "v182", "v183", "v184", "v185", "v186", "v187", "v188",
      "v189", "v190", "v191", "v192", "v193", "v194", "v195", "v196", "v197", "v198", "v199", "v200", "v201", "v202", "v203", "v204",
      "v205", "v206", "v207", "v208", "v209", "v210", "v211", "v212", "v213", "v214", "v215", "v216", "v217", "v218", "v219", "v220",
      "v221", "v222", "v223", "v224", "v225", "v226", "v227", "v228", "v229", "v230", "v231", "v232", "v233", "v234", "v235", "v236",
      "v237", "v238", "v239", "v240", "v241", "v242", "v243", "v244", "v245", "v246", "v247", "v248", "v249", "v250", "v251", "v252",
      "v253", "v254", "v255", "a0", "a1", "a2", "a3", "a4", "a5", "a6", "a7", "a8", "a9", "a10", "a11", "a12",
      "a13", "a14", "a15", "a16", "a17", "a18", "a19", "a20", "a21", "a22", "a23", "a24", "a25", "a26", "a27", "a28",
      "a29", "a30", "a31", "a32", "a33", "a34", "a35", "a36", "a37", "a38", "a39", "a40", "a41", "a42", "a43", "a44",
      "a45", "a46", "a47", "a48", "a49", "a50", "a51", "a52", "a53", "a54", "a55", "a56", "a57", "a58", "a59", "a60",
      "a61", "a62", "a63", "a64", "a65", "a66", "a67", "a68", "a69", "a70", "a71", "a72", "a73", "a74", "a75", "a76",
      "a77", "a78", "a79", "a80", "a81", "a82", "a83", "a84", "a85", "a86", "a87", "a88", "a89", "a90", "a91", "a92",
      "a93", "a94", "a95", "a96", "a97", "a98", "a99", "a100", "a101", "a102", "a103", "a104", "a105", "a106", "a107", "a108",
      "a109", "a110", "a111", "a112", "a113", "a114", "a115", "a116", "a117", "a118", "a119", "a120", "a121", "a122", "a123", "a124",
      "a125", "a126", "a127", "a128", "a129", "a130", "a131", "a132", "a133", "a134", "a135", "a136", "a137", "a138", "a139", "a140",
      "a141", "a142", "a143", "a144", "a145", "a146", "a147", "a148", "a149", "a150", "a151", "a152", "a153", "a154", "a155", "a156",
      "a157", "a158", "a159", "a160", "a161", "a162", "a163", "a164", "a165", "a166", "a167", "a168", "a169", "a170", "a171", "a172",
      "a173", "a174", "a175", "a176", "a177", "a178", "a179", "a180", "a181", "a182", "a183", "a184", "a185", "a186", "a187", "a188",
      "a189", "a190", "a191", "a192", "a193", "a194", "a195", "a196", "a197", "a198", "a199", "a200", "a201", "a202", "a203", "a204",
      "a205", "a206", "a207", "a208", "a209", "a210", "a211", "a212", "a213", "a214", "a215", "a216", "a217", "a218", "a219", "a220",
      "a221", "a222", "a223", "a224", "a225", "a226", "a227", "a228", "a229", "a230", "a231", "a232", "a233", "a234", "a235", "a236",
      "a237", "a238", "a239", "a240", "a241", "a242", "a243", "a244", "a245", "a246", "a247", "a248", "a249", "a250", "a251", "a252",
      "a253", "a254", "a255", "s33", "s34", "s35", "s44", "s45", "s46", "s47", "s48", "s49", "s50", "s52", "s56", "s57",
      "s59", "scc", "vcc", "memory");
}

extern "C" void kernel_launch(int rank, int nranks, ncclComm_t comm, hipStream_t stream,
                              void* const* d_in, const int* in_sizes, int n_in,
                              void* d_out, int out_size, const int* in_dtypes, int out_dtype,
                              void* const* d_out_all, void* const* d_ws_all, size_t ws_bytes) {
  (void)comm; (void)in_dtypes; (void)out_dtype; (void)d_out_all;
  if (nranks != (int)kRanks || (unsigned)rank >= kRanks || n_in != 2 || in_sizes[0] != (int)(kK * kM) ||
      in_sizes[1] != (int)(kK * kN) || out_size != (int)(kM * kN) || ws_bytes < (size_t)kCtl + kCtlBytes) {
    fprintf(stderr, "a2a_gemm_silu: unexpected problem shape\n");
    return;
  }
  Params p;
  p.x = (const float*)d_in[0]; p.w = (const float*)d_in[1]; p.out = (float*)d_out;
  for (unsigned q = 0; q < kRanks; ++q) p.ws[q] = (char*)d_ws_all[((unsigned)rank - 1 - q) & 7u];
  p.rank = (unsigned)rank; p.pad = 0;
  hipLaunchKernelGGL(a2a_gemm_silu_kernel, dim3(256), dim3(256), 0, stream, p);
}
```
